```python
import numpy as np

kernel_name = "gpu_gemm_rs_m4096_k4096_n4096_w8a8_dqsilu"


ABSMAX_RTOL = 0.001

COMPUTE_DTYPE = "int8"
EPILOGUE = "dequant_silu"
IN_QUANT = "int8"
OUT_QUANT = None
IN_DTYPES = {"x": "int8", "w_mat": "int8",
             "scale_x": "float32", "scale_w": "float32"}
WORLD_SIZE = 8

def setup_inputs():

    x = np.random.randint(-127, 128, size=(4096, 4096)).astype(np.int8)
    w_mat = np.random.randint(-127, 128, size=(4096, 4096)).astype(np.int8)
    scale_x = (0.02 * (1.0 + np.random.rand(1))).astype(np.float32)
    scale_w = (0.02 * (1.0 + np.random.rand(1))).astype(np.float32)
    return {"x": x, "w_mat": w_mat, "scale_x": scale_x, "scale_w": scale_w}

def reference(x, w_mat, scale_x, scale_w):
    acc = x.astype(np.float64) @ w_mat.astype(np.float64)
    y = (acc.astype(np.float32) * (scale_x * scale_w)).astype(np.float32)
    return (y / (1.0 + np.exp(-np.clip(y, -60.0, 60.0)))).astype(np.float32)
```

```cpp
#include <hip/hip_runtime.h>
#include <rccl/rccl.h>
#include <cstdio>

#define HIP_CHECK(expr) do { hipError_t _e = (expr); if (_e != hipSuccess) { \
    fprintf(stderr, "HIP error %s at %s:%d\n", hipGetErrorString(_e), __FILE__, __LINE__); \
    return; } } while (0)

#define GAS __attribute__((address_space(1)))
#define LAS __attribute__((address_space(3)))
typedef unsigned u32x4 __attribute__((ext_vector_type(4)));
typedef int i32x4 __attribute__((ext_vector_type(4)));
typedef int i32x16 __attribute__((ext_vector_type(16)));
typedef unsigned long long u64;

namespace {

constexpr int NR = 8;
constexpr int NDIM = 4096;
constexpr int KS = 512;
constexpr int MS = 512;
constexpr int NSLAB = NDIM / 32;
constexpr int NKS = KS / 32;
constexpr int NUNIT = NR * NKS;
constexpr int N_WST = 64, N_XP = 64, N_STAGE = N_WST + N_XP;
constexpr int GRID = N_STAGE + NSLAB;
constexpr int THREADS = 576;

constexpr size_t OFF_CNT = 0;
constexpr size_t OFF_TMO = OFF_CNT + GRID * 128;
constexpr size_t OFF_WFLAG = OFF_TMO + 128;
constexpr size_t OFF_XFLAG = OFF_WFLAG + 32 * 128;
constexpr size_t OFF_STAMP = 64 << 10;
constexpr size_t STAGE_BYTES = 2u << 20;
constexpr size_t OFF_X = 1u << 20;
constexpr size_t OFF_W = OFF_X + 2 * STAGE_BYTES;
constexpr size_t WS_NEED = OFF_W + 2 * STAGE_BYTES;
static_assert(OFF_XFLAG + 256 <= OFF_STAMP, "layout");

constexpr int LDS_B = NUNIT * 1024;
constexpr int LDS_CTL = LDS_B;
constexpr int LDS_BYTES = LDS_B + 64;

constexpr u64 DEADLINE_TICKS = 50000000ull;
#define DEPTH 3

struct Params {
  const char* x;
  const char* w;
  const float* sx;
  const float* sw;
  float* out;
  char* ws[NR];
  int rank;
  int pad;
};

__device__ __forceinline__ u64 rt() { return __builtin_amdgcn_s_memrealtime(); }
template <int N> __device__ __forceinline__ void wait_vm() { asm volatile("s_waitcnt vmcnt(%0)" :: "i"(N) : "memory"); }
__device__ __forceinline__ void store_wt(void* p, u32x4 v) {
  asm volatile("global_store_dwordx4 %0, %1, off sc0 sc1\n\ts_nop 1" :: "v"(p), "v"(v) : "memory");
}
__device__ __forceinline__ void lds_write(unsigned addr, unsigned v) {
  asm volatile("ds_write_b32 %0, %1" :: "v"(addr), "v"(v) : "memory");
}
__device__ __forceinline__ unsigned lds_read(unsigned addr) {
  unsigned v;
  asm volatile("ds_read_b32 %0, %1\n\ts_waitcnt lgkmcnt(0)" : "=v"(v) : "v"(addr) : "memory");
  return v;
}
__device__ __forceinline__ unsigned ld_sys(const void* p) {
  return __hip_atomic_load((const GAS unsigned*)p, __ATOMIC_RELAXED, __HIP_MEMORY_SCOPE_SYSTEM);
}
__device__ __forceinline__ unsigned ld_agent(const void* p) {
  return __hip_atomic_load((const GAS unsigned*)p, __ATOMIC_RELAXED, __HIP_MEMORY_SCOPE_AGENT);
}
__device__ __forceinline__ void st_sys(void* p, unsigned v) {
  __hip_atomic_store((GAS unsigned*)p, v, __ATOMIC_RELAXED, __HIP_MEMORY_SCOPE_SYSTEM);
}
__device__ __forceinline__ void st_agent(void* p, unsigned v) {
  __hip_atomic_store((GAS unsigned*)p, v, __ATOMIC_RELAXED, __HIP_MEMORY_SCOPE_AGENT);
}
__device__ __forceinline__ unsigned perm(unsigned hi, unsigned lo, unsigned sel) { return __builtin_amdgcn_perm(hi, lo, sel); }

#define STAMP(i) do { } while (0)

__device__ __forceinline__ bool wait_flags(const char* base, int n, int lane, unsigned call, char* tmo) {
  const u64 t0 = rt();
  const char* q = base + (lane < n ? lane : 0) * 4;
  for (;;) {
    const unsigned v = ld_sys(q);
    if (__all(v >= call)) return true;
    if (ld_agent(tmo) == call || rt() - t0 > DEADLINE_TICKS) {
      st_agent(tmo, call);
      return false;
    }
    __builtin_amdgcn_s_sleep(2);
  }
}

__global__ void __launch_bounds__(THREADS) gemm_ag_kernel(Params p) {
  extern __shared__ __attribute__((aligned(16))) char lds[];
  const int tid = threadIdx.x;
  const int wave = __builtin_amdgcn_readfirstlane(tid >> 6);
  const int lane = tid & 63;
  const int bid = blockIdx.x;
  char* const myws = p.ws[p.rank];
  const unsigned ctl = (unsigned)(size_t)(LAS char*)lds + LDS_CTL;

  unsigned cnt0 = 0;
  if (tid == 0) cnt0 = ld_agent(myws + OFF_CNT + bid * 128);
  if (wave == 0) STAMP(0);

  if (bid < N_WST) {
    const int G = bid >> 1, ks = (bid & 1) * 8 + wave;
    unsigned r[16];
    if (wave < 8) {
      const char* src = p.w + (size_t)(ks * 32 + (lane >> 5) * 16) * NDIM + G * 128 + (lane & 31) * 4;
#pragma unroll
      for (int i = 0; i < 16; ++i) r[i] = *(const GAS unsigned*)(src + (size_t)i * NDIM);
    }
    if (tid == 0) { lds_write(ctl, cnt0 + 1); st_agent(myws + OFF_CNT + bid * 128, cnt0 + 1); asm volatile("s_waitcnt lgkmcnt(0)" ::: "memory"); }
    __syncthreads();
    const unsigned call = __builtin_amdgcn_readfirstlane(lds_read(ctl));
    if (wave < 8) {
      u32x4 c[4];
#pragma unroll
      for (int g = 0; g < 4; ++g) {
        const unsigned lo01 = perm(r[4 * g + 1], r[4 * g], 0x05010400u), hi01 = perm(r[4 * g + 1], r[4 * g], 0x07030602u);
        const unsigned lo23 = perm(r[4 * g + 3], r[4 * g + 2], 0x05010400u), hi23 = perm(r[4 * g + 3], r[4 * g + 2], 0x07030602u);
        c[0][g] = perm(lo23, lo01, 0x05040100u);
        c[1][g] = perm(lo23, lo01, 0x07060302u);
        c[2][g] = perm(hi23, hi01, 0x05040100u);
        c[3][g] = perm(hi23, hi01, 0x07060302u);
      }
      const int slab = G * 4 + ((lane >> 3) & 3);
      char* dst = myws + OFF_W + (size_t)(call & 1) * STAGE_BYTES + (size_t)(slab * NKS + ks) * 1024 + ((lane & 7) + (lane >> 5) * 32) * 16;
#pragma unroll
      for (int q = 0; q < 4; ++q) store_wt(dst + q * 128, c[q]);
    }
    wait_vm<0>();
    __syncthreads();
    if (tid < NR) st_sys(p.ws[tid] + OFF_WFLAG + G * 128 + (p.rank * 2 + (bid & 1)) * 4, call);
    if (wave == 0) STAMP(1);
    return;
  }
  if (bid < N_STAGE) {
    const int idx = bid - N_WST;
    const int dst_rank = (p.rank + (idx >> 3)) & (NR - 1), part = idx & 7;
    const int rtile = part * 2 + (wave & 1), ks0 = (wave >> 1) * 4;
    u32x4 v[4];
    if (wave < 8) {
      const char* src = p.x + (size_t)(dst_rank * MS + rtile * 32 + (lane & 31)) * KS + ks0 * 32 + (lane >> 5) * 16;
#pragma unroll
      for (int i = 0; i < 4; ++i) v[i] = *(const GAS u32x4*)(src + i * 32);
    }
    if (tid == 0) { lds_write(ctl, cnt0 + 1); st_agent(myws + OFF_CNT + bid * 128, cnt0 + 1); asm volatile("s_waitcnt lgkmcnt(0)" ::: "memory"); }
    __syncthreads();
    const unsigned call = __builtin_amdgcn_readfirstlane(lds_read(ctl));
    char* const dws = p.ws[dst_rank];
    if (wave < 8) {
      char* dst = dws + OFF_X + (size_t)(call & 1) * STAGE_BYTES + (size_t)((p.rank * NKS + ks0) * 16 + rtile) * 1024 + lane * 16;
#pragma unroll
      for (int i = 0; i < 4; ++i) store_wt(dst + i * 16384, v[i]);
    }
    wait_vm<0>();
    __syncthreads();
    if (tid == 0) st_sys(dws + OFF_XFLAG + (p.rank * 8 + part) * 4, call);
    if (wave == 0) STAMP(1);
    return;
  }

  const int slab = bid - N_STAGE;
  if (tid == 0) {
    lds_write(ctl, cnt0 + 1); lds_write(ctl + 4, 0); lds_write(ctl + 8, 0); lds_write(ctl + 12, 0);
    st_agent(myws + OFF_CNT + bid * 128, cnt0 + 1);
    asm volatile("s_waitcnt lgkmcnt(0)" ::: "memory");
  }
  __syncthreads();
  const unsigned call = __builtin_amdgcn_readfirstlane(lds_read(ctl));
  char* const tmo = myws + OFF_TMO;
  const int j0 = (slab + (slab >> 3)) & (NR - 1);

  if (wave == 8) {
    const bool ok = wait_flags(myws + OFF_WFLAG + (slab >> 2) * 128, 16, lane, call, tmo);
    STAMP(1);
    if (!ok) {
      lds_write(ctl + 12, 1);
      lds_write(ctl + 4, NUNIT);
      return;
    }
    const char* src[NR];
#pragma unroll
    for (int i = 0; i < NR; ++i)
      src[i] = p.ws[(j0 + i) & (NR - 1)] + OFF_W + (size_t)(call & 1) * STAGE_BYTES + (size_t)slab * NKS * 1024 + lane * 16;
    for (int ks = 0; ks < NKS; ++ks) {
#pragma unroll
      for (int i = 0; i < NR; ++i) {
        __builtin_amdgcn_global_load_lds((const GAS unsigned*)(src[i] + ks * 1024), (LAS unsigned*)(lds + (ks * NR + i) * 1024), 16, 0, 17);
        wait_vm<DEPTH>();
        const int landed = ks * NR + i + 1 - DEPTH;
        if (landed > 0) lds_write(ctl + 4, landed);
      }
    }
    wait_vm<0>();
    lds_write(ctl + 4, NUNIT);
    STAMP(2);
    return;
  }

  if (wave == 0) {
    const bool ok = wait_flags(myws + OFF_XFLAG, 64, lane, call, tmo);
    __builtin_amdgcn_fence(__ATOMIC_ACQUIRE, "");
    asm volatile("s_waitcnt vmcnt(0)" ::: "memory");
    if (!ok || ld_agent(tmo) == call) lds_write(ctl + 12, 1);
    lds_write(ctl + 8, 1);
    STAMP(3);
  } else {
    while (lds_read(ctl + 8) == 0) __builtin_amdgcn_s_sleep(4);
  }
  asm volatile("" ::: "memory");

  const char* xa = myws + OFF_X + (size_t)(call & 1) * STAGE_BYTES + wave * 2048 + lane * 16;
  auto a_addr = [&](int t) { return xa + (size_t)((((j0 + t) & (NR - 1)) * NKS + (t >> 3)) << 14); };
  constexpr int PF = 4;
  i32x4 a[PF][2];
#pragma unroll
  for (int i = 0; i < PF; ++i) {
    const char* q = a_addr(i);
    a[i][0] = *(const GAS i32x4*)q;
    a[i][1] = *(const GAS i32x4*)(q + 1024);
  }
  i32x16 acc0 = {}, acc1 = {};
  unsigned landed = 0;
  for (int t0 = 0; t0 < NUNIT; t0 += PF) {
#pragma unroll
    for (int i = 0; i < PF; ++i) {
      const int t = t0 + i;
      while (landed <= (unsigned)t) {
        landed = __builtin_amdgcn_readfirstlane(lds_read(ctl + 4));
        if (landed <= (unsigned)t) __builtin_amdgcn_s_sleep(1);
      }
      const i32x4 b = *(const LAS i32x4*)(lds + t * 1024 + lane * 16);
      acc0 = __builtin_amdgcn_mfma_i32_32x32x32_i8(a[i][0], b, acc0, 0, 0, 0);
      acc1 = __builtin_amdgcn_mfma_i32_32x32x32_i8(a[i][1], b, acc1, 0, 0, 0);
      const int tn = (t + PF < NUNIT) ? t + PF : t;
      const char* q = a_addr(tn);
      a[i][0] = *(const GAS i32x4*)q;
      a[i][1] = *(const GAS i32x4*)(q + 1024);
    }
  }
  if (wave == 0) STAMP(4);

  const bool bad = lds_read(ctl + 12) != 0;
  const float scale = p.sx[0] * p.sw[0];
  const int c31 = lane & 31;
  float* o = p.out + (size_t)(wave * 64 + (lane >> 5) * 4) * NDIM + slab * 32 + (c31 & 7) * 4 + (c31 >> 3);
#pragma unroll
  for (int h = 0; h < 2; ++h) {
#pragma unroll
    for (int r = 0; r < 16; ++r) {
      const int acc = h == 0 ? acc0[r] : acc1[r];
      const float y = (float)acc * scale;
      const float z = fminf(fmaxf(y, -60.0f), 60.0f);
      float v = __fdividef(y, 1.0f + __expf(-z));
      if (bad) v = __builtin_nanf("");
      o[(size_t)(h * 32 + (r & 3) + 8 * (r >> 2)) * NDIM] = v;
    }
  }
}

}

extern "C" void kernel_launch(int rank, int nranks, ncclComm_t comm, hipStream_t stream,
                              void* const* d_in, const int* in_sizes, int n_in,
                              void* d_out, int out_size, const int* in_dtypes, int out_dtype,
                              void* const* d_out_all, void* const* d_ws_all, size_t ws_bytes) {
  (void)comm; (void)in_sizes; (void)in_dtypes; (void)out_dtype; (void)d_out_all;
  if (nranks != NR || n_in != 4 || out_size != MS * NDIM || ws_bytes < WS_NEED || rank < 0 || rank >= NR) {
    fprintf(stderr, "kernel_launch: unexpected configuration\n");
    return;
  }
  static bool attr_set[NR] = {};
  if (!attr_set[rank]) {
    HIP_CHECK(hipFuncSetAttribute((const void*)gemm_ag_kernel, hipFuncAttributeMaxDynamicSharedMemorySize, LDS_BYTES));
    attr_set[rank] = true;
  }
  Params p;
  p.x = (const char*)d_in[0];
  p.w = (const char*)d_in[1];
  p.sx = (const float*)d_in[2];
  p.sw = (const float*)d_in[3];
  p.out = (float*)d_out;
  for (int i = 0; i < NR; ++i) p.ws[i] = (char*)d_ws_all[i];
  p.rank = rank;
  p.pad = 0;
  hipLaunchKernelGGL(gemm_ag_kernel, dim3(GRID), dim3(THREADS), LDS_BYTES, stream, p);
  HIP_CHECK(hipGetLastError());
}
```

```cpp
#include <hip/hip_runtime.h>
#include <rccl/rccl.h>
#include <cstdio>

#define HIP_CHECK(expr) do { hipError_t _e = (expr); if (_e != hipSuccess) { \
    fprintf(stderr, "HIP error %s at %s:%d\n", hipGetErrorString(_e), __FILE__, __LINE__); \
    return; } } while (0)

#define GAS __attribute__((address_space(1)))
#define LAS __attribute__((address_space(3)))
typedef unsigned u32x4 __attribute__((ext_vector_type(4)));
typedef int i32x4 __attribute__((ext_vector_type(4)));
typedef int i32x16 __attribute__((ext_vector_type(16)));
typedef unsigned long long u64;

#define DEPTH 6
#define PACE16 448
#define PFD 4
#define XDELAY 0
#define GODELAY 0
#define NSW 16
#define DELTA 80
#define PDEPTH 6
#define OUT_AUX 17
#define XSLEEP 8
#define LPRIO 0

namespace {

constexpr int NR = 8;
constexpr int NDIM = 4096;
constexpr int KS = 512;
constexpr int MS = 512;
constexpr int NSLAB = NDIM / 32;
constexpr int NKS = KS / 32;
constexpr int NUNIT = NR * NKS;
constexpr int N_WST = 64, N_STAGE = 128;
constexpr int GRID = N_STAGE + NSLAB;
constexpr int THREADS = 576;

constexpr size_t OFF_CNT = 0;
constexpr size_t OFF_TMO = OFF_CNT + GRID * 128;
constexpr size_t OFF_WCNT = OFF_TMO + 128;
constexpr size_t OFF_GO = OFF_WCNT + 8 * 128;
constexpr size_t OFF_XFLAG = OFF_GO + 8 * 128;
constexpr size_t OFF_STAMP = 64 << 10;
constexpr size_t STAGE_BYTES = 2u << 20;
constexpr size_t OFF_X = 1u << 20;
constexpr size_t OFF_W = OFF_X + 2 * STAGE_BYTES;
constexpr size_t WS_NEED = OFF_W + 2 * STAGE_BYTES;
static_assert(OFF_XFLAG + 8 * 256 <= OFF_STAMP, "layout");

constexpr int LDS_B = NUNIT * 1024;
constexpr int LDS_CTL = LDS_B;
constexpr int LDS_BYTES = LDS_B + 64;

constexpr u64 DEADLINE_TICKS = 50000000ull;

struct Params {
  const char* x;
  const char* w;
  const float* sx;
  const float* sw;
  float* out;
  char* myws;
  char* ws[NR];
  int rank;
  int pad;
};

__device__ __forceinline__ u64 rt() { return __builtin_amdgcn_s_memrealtime(); }
template <int N> __device__ __forceinline__ void wait_vm() { asm volatile("s_waitcnt vmcnt(%0)" :: "i"(N) : "memory"); }
__device__ __forceinline__ void store_wt(void* p, u32x4 v) {
  asm volatile("global_store_dwordx4 %0, %1, off sc0 sc1\n\ts_nop 1" :: "v"(p), "v"(v) : "memory");
}
__device__ __forceinline__ void lds_write(unsigned addr, unsigned v) {
  asm volatile("ds_write_b32 %0, %1" :: "v"(addr), "v"(v) : "memory");
}
template <int D> __device__ __forceinline__ void drain(unsigned addr) {
  if constexpr (D > 0) {
    wait_vm<D>();
    lds_write(addr, NUNIT - D);
    drain<D - 1>(addr);
  }
}
__device__ __forceinline__ unsigned lds_read(unsigned addr) {
  unsigned v;
  asm volatile("ds_read_b32 %0, %1\n\ts_waitcnt lgkmcnt(0)" : "=v"(v) : "v"(addr) : "memory");
  return v;
}
__device__ __forceinline__ unsigned ld_sys(const void* p) {
  return __hip_atomic_load((const GAS unsigned*)p, __ATOMIC_RELAXED, __HIP_MEMORY_SCOPE_SYSTEM);
}
__device__ __forceinline__ unsigned ld_agent(const void* p) {
  return __hip_atomic_load((const GAS unsigned*)p, __ATOMIC_RELAXED, __HIP_MEMORY_SCOPE_AGENT);
}
__device__ __forceinline__ void st_sys(void* p, unsigned v) {
  __hip_atomic_store((GAS unsigned*)p, v, __ATOMIC_RELAXED, __HIP_MEMORY_SCOPE_SYSTEM);
}
__device__ __forceinline__ void st_agent(void* p, unsigned v) {
  __hip_atomic_store((GAS unsigned*)p, v, __ATOMIC_RELAXED, __HIP_MEMORY_SCOPE_AGENT);
}
__device__ __forceinline__ unsigned perm(unsigned hi, unsigned lo, unsigned sel) { return __builtin_amdgcn_perm(hi, lo, sel); }
__device__ __forceinline__ void wait_until(unsigned t) {
  while ((int)((unsigned)rt() - t) < 0) __builtin_amdgcn_s_sleep(1);
}

#define STAMP(i) do { } while (0)

template <int SLEEP>
__device__ __forceinline__ bool wait_flags(const char* base, int n, int lane, int stride, unsigned target, unsigned call, char* tmo) {
  const u64 t0 = rt();
  const char* q = base + (lane < n ? lane : 0) * stride;
  for (unsigned it = 1;; ++it) {
    const unsigned v = ld_sys(q);
    if (__all(v >= target)) return true;
    if ((it & 63) == 0 && (ld_agent(tmo) == call || rt() - t0 > DEADLINE_TICKS)) { st_agent(tmo, call); return false; }
    __builtin_amdgcn_s_sleep(SLEEP);
  }
}

__device__ __forceinline__ const char* frag_src(u64 tab, int t, int lane) {
  const unsigned lo = __builtin_amdgcn_readlane((unsigned)tab, t & 7), hi = __builtin_amdgcn_readlane((unsigned)(tab >> 32), t & 7);
  return (const char*)(((u64)hi << 32) | lo) + (t >> 3) * 1024 + lane * 16;
}

__global__ void __launch_bounds__(THREADS) gemm_ag_kernel(Params p) {
  extern __shared__ __attribute__((aligned(16))) char lds[];
  const int tid = threadIdx.x;
  const int wave = __builtin_amdgcn_readfirstlane(tid >> 6);
  const int lane = tid & 63;
  const int bid = blockIdx.x;
  char* const myws = p.myws;
  char* const tmo = myws + OFF_TMO;
  const unsigned ctl = (unsigned)(size_t)(LAS char*)lds + LDS_CTL;

  char* const cntp = myws + OFF_CNT + bid * 128;
  unsigned cnt_v = ld_agent(cntp);
  if (wave == 0) STAMP(0);
#define GET_CALL() ({ asm volatile("" : "+v"(cnt_v) :: "memory"); (unsigned)__builtin_amdgcn_readfirstlane(cnt_v) + 1; })
  const unsigned t_start = (unsigned)rt();

  if (bid < N_WST) {
    const int G = bid >> 1, kh = bid & 1;
    const int pslab = bid + (wave & 1) * 64, pj0 = (pslab + (pslab >> 3)) & (NR - 1);
    u64 tab = (u64)p.ws[wave == 2 ? (lane & (NR - 1)) : ((pj0 + lane) & (NR - 1))];
    unsigned call;
    if (wave < 8) {
      const int ks = kh * 8 + wave;
      unsigned r[16];
      u32x4 v[4];
      const char* src = p.w + (size_t)(ks * 32 + (lane >> 5) * 16) * NDIM + G * 128 + (lane & 31) * 4;
#pragma unroll
      for (int i = 0; i < 16; ++i) r[i] = *(const GAS unsigned*)(src + (size_t)i * NDIM);
      call = GET_CALL();
#pragma unroll
      for (int g = 0; g < 4; ++g) {
        const unsigned lo01 = perm(r[4 * g + 1], r[4 * g], 0x05010400u), hi01 = perm(r[4 * g + 1], r[4 * g], 0x07030602u);
        const unsigned lo23 = perm(r[4 * g + 3], r[4 * g + 2], 0x05010400u), hi23 = perm(r[4 * g + 3], r[4 * g + 2], 0x07030602u);
        v[0][g] = perm(lo23, lo01, 0x05040100u);
        v[1][g] = perm(lo23, lo01, 0x07060302u);
        v[2][g] = perm(hi23, hi01, 0x05040100u);
        v[3][g] = perm(hi23, hi01, 0x07060302u);
      }
      const int slab = G * 4 + ((lane >> 3) & 3);
      char* dst = myws + OFF_W + (size_t)(call & 1) * STAGE_BYTES + (size_t)(slab * NKS + ks) * 1024 + ((lane & 7) + (lane >> 5) * 32) * 16;
#pragma unroll
      for (int q = 0; q < 4; ++q) store_wt(dst + q * 128, v[q]);
      wait_vm<0>();
    } else {
      call = GET_CALL();
    }
    asm volatile("s_barrier" ::: "memory");
    if (tid == 0) {
      __hip_atomic_fetch_add((GAS unsigned*)(myws + OFF_WCNT + (bid & 7) * 128), 1u, __ATOMIC_RELAXED, __HIP_MEMORY_SCOPE_AGENT);
      st_agent(cntp, call);
    }
    if (wave == 0) STAMP(1);
    if (bid == 0 && wave == 2) {
      const bool ok = wait_flags<1>(myws + OFF_WCNT, 8, lane, 128, call * (N_WST / 8), call, tmo);
      wait_until(t_start + GODELAY);
      if (ok) st_sys((char*)tab + OFF_GO + (lane >> 3) * 128 + p.rank * 4, call);
      STAMP(2);
    }
    if (wave < 2 && NSW < NUNIT) {
      tab += OFF_W + (size_t)(call & 1) * STAGE_BYTES + (size_t)pslab * NKS * 1024;
      if (!wait_flags<1>(myws + OFF_GO + (bid & 7) * 128, 8, lane, 4, call, call, tmo)) return;
      const unsigned t0 = (unsigned)rt();
      if (wave == 0) STAMP(3);
      for (int t = NSW; t < NUNIT; ++t) {
        if (((pj0 + t) & (NR - 1)) == p.rank) continue;
        const char* s = frag_src(tab, t, lane);
        wait_until(t0 + ((unsigned)(t * PACE16) >> 4));
        __builtin_amdgcn_global_load_lds((const GAS unsigned*)s, (LAS unsigned*)(lds + ((wave * 16 + (t & 15)) << 10)), 16, 0, 0);
        wait_vm<PDEPTH>();
      }
      wait_vm<0>();
      if (wave == 0) STAMP(4);
    }
    return;
  }
  if (bid < N_STAGE) {
    const int idx = bid - N_WST;
    const int dst_rank = (p.rank + (idx >> 3)) & (NR - 1), part = idx & 7;
    char* const dws = p.ws[dst_rank];
    unsigned call = 0;
    if (wave < 8) {
      const int rtile = part * 2 + (wave & 1), ks0 = (wave >> 1) * 4;
      u32x4 v[4];
      const char* src = p.x + (size_t)(dst_rank * MS + rtile * 32 + (lane & 31)) * KS + ks0 * 32 + (lane >> 5) * 16;
#pragma unroll
      for (int i = 0; i < 4; ++i) v[i] = *(const GAS u32x4*)(src + i * 32);
      call = GET_CALL();
      char* xdst = dws + OFF_X + (size_t)(call & 1) * STAGE_BYTES + (size_t)((p.rank * NKS + ks0) * 16 + rtile) * 1024 + lane * 16;
      wait_until(t_start + XDELAY);
#pragma unroll
      for (int i = 0; i < 4; ++i) store_wt(xdst + i * 16384, v[i]);
    }
    wait_vm<0>();
    asm volatile("s_barrier" ::: "memory");
    if (tid < 8) st_sys(dws + OFF_XFLAG + tid * 256 + (p.rank * 8 + part) * 4, call);
    if (tid == 0) st_agent(cntp, call);
    if (wave == 0) STAMP(2);
    return;
  }

  const int slab = bid - N_STAGE;
  const int j0 = (slab + (slab >> 3)) & (NR - 1);
  u64 tab = 0;
  if (wave == 8) tab = (u64)p.ws[(j0 + lane) & (NR - 1)];
  const unsigned call = GET_CALL();
  if (tid == 0) {
    lds_write(ctl + 4, 0); lds_write(ctl + 8, 0); lds_write(ctl + 12, 0); lds_write(ctl + 16, 0);
    asm volatile("s_waitcnt lgkmcnt(0)" ::: "memory");
  }
  asm volatile("s_barrier" ::: "memory");
  if (tid == 0) st_agent(cntp, call);

  if (wave == 8) {
    tab += OFF_W + (size_t)(call & 1) * STAGE_BYTES + (size_t)slab * NKS * 1024;
    const bool ok = wait_flags<1>(myws + OFF_GO + (bid & 7) * 128, 8, lane, 4, call, call, tmo);
    const unsigned t0 = (unsigned)rt();
    STAMP(1);
    if (!ok) {
      lds_write(ctl + 12, 1);
      lds_write(ctl + 16, 1);
      lds_write(ctl + 4, NUNIT);
      return;
    }
    lds_write(ctl + 16, 1);
    for (int t = 0; t < NSW; ++t) {
      const char* s = frag_src(tab, t, lane);
      wait_until(t0 + ((unsigned)(t * PACE16) >> 4));
      __builtin_amdgcn_global_load_lds((const GAS unsigned*)s, (LAS unsigned*)(lds + t * 1024), 16, 0, 17);
      wait_vm<DEPTH>();
      if (t + 1 - DEPTH > 0) lds_write(ctl + 4, t + 1 - DEPTH);
    }
    while (lds_read(ctl + 8) == 0) __builtin_amdgcn_s_sleep(1);
    for (int t = NSW; t < NUNIT; ++t) {
      const char* s = frag_src(tab, t, lane);
      const unsigned ts = (unsigned)(t * PACE16) >> 4, te = (unsigned)((NUNIT - 1) * PACE16) >> 4;
      wait_until(t0 + (ts + DELTA < te ? ts + DELTA : te));
      __builtin_amdgcn_global_load_lds((const GAS unsigned*)s, (LAS unsigned*)(lds + t * 1024), 16, 0, 0);
      wait_vm<DEPTH>();
      if (t + 1 - DEPTH > 0) lds_write(ctl + 4, t + 1 - DEPTH);
    }
    drain<DEPTH - 1>(ctl + 4);
    wait_vm<0>();
    lds_write(ctl + 4, NUNIT);
    STAMP(2);
    return;
  }

  float scale0 = p.sx[0] * p.sw[0];
  unsigned out_lo = (unsigned)(size_t)p.out, out_hi = (unsigned)((size_t)p.out >> 32);
  asm volatile("" : "+v"(scale0), "+v"(out_lo), "+v"(out_hi));
  if (wave == 0) {
    const bool ok = wait_flags<XSLEEP>(myws + OFF_XFLAG + (bid & 7) * 256, 64, lane, 4, call, call, tmo);
    while (lds_read(ctl + 16) == 0) __builtin_amdgcn_s_sleep(1);
    STAMP(9);
    __builtin_amdgcn_fence(__ATOMIC_ACQUIRE, "");
    asm volatile("s_waitcnt vmcnt(0)" ::: "memory");
    if (!ok || ld_agent(tmo) == call) lds_write(ctl + 12, 1);
    lds_write(ctl + 8, 1);
    STAMP(3);
  } else {
    while (lds_read(ctl + 8) == 0) __builtin_amdgcn_s_sleep(4);
  }
  asm volatile("" ::: "memory");

  const char* xa = myws + OFF_X + (size_t)(call & 1) * STAGE_BYTES + wave * 2048 + lane * 16;
  auto a_addr = [&](int t) { return xa + (size_t)((((j0 + t) & (NR - 1)) * NKS + (t >> 3)) << 14); };
  constexpr int PF = PFD;
  static_assert(NUNIT % PF == 0, "prefetch depth");
  i32x4 a[PF][2];
#pragma unroll
  for (int i = 0; i < PF; ++i) {
    const char* q = a_addr(i);
    a[i][0] = *(const GAS i32x4*)q;
    a[i][1] = *(const GAS i32x4*)(q + 1024);
  }
  i32x16 acc0 = {}, acc1 = {};
  unsigned landed = 0;
  auto step = [&](int t, int i, bool prefetch) {
    while (landed <= (unsigned)t) {
      landed = __builtin_amdgcn_readfirstlane(lds_read(ctl + 4));
      if (landed <= (unsigned)t) __builtin_amdgcn_s_sleep(1);
    }
    const i32x4 b = *(const LAS i32x4*)(lds + t * 1024 + lane * 16);
    acc0 = __builtin_amdgcn_mfma_i32_32x32x32_i8(a[i][0], b, acc0, 0, 0, 0);
    acc1 = __builtin_amdgcn_mfma_i32_32x32x32_i8(a[i][1], b, acc1, 0, 0, 0);
    if (prefetch) {
      const char* q = a_addr(t + PF);
      a[i][0] = *(const GAS i32x4*)q;
      a[i][1] = *(const GAS i32x4*)(q + 1024);
    }
  };
  for (int t0 = 0; t0 < NUNIT - PF; t0 += PF) {
#pragma unroll
    for (int i = 0; i < PF; ++i) step(t0 + i, i, true);
  }
#pragma unroll
  for (int i = 0; i < PF; ++i) step(NUNIT - PF + i, i, false);
  if (wave == 0) STAMP(4);
  STAMP(16 + wave);

  const float scale = lds_read(ctl + 12) != 0 ? __builtin_nanf("") : scale0;
  const int c31 = lane & 31;
  float* outp = (float*)(((size_t)(unsigned)__builtin_amdgcn_readfirstlane(out_hi) << 32) | (unsigned)__builtin_amdgcn_readfirstlane(out_lo));
  const __amdgpu_buffer_rsrc_t ors = __builtin_amdgcn_make_buffer_rsrc(outp, 0, MS * NDIM * 4, 0x00020000);
  const unsigned voff = (unsigned)(((wave * 64 + (lane >> 5) * 4) * NDIM + slab * 32 + (c31 & 7) * 4 + (c31 >> 3)) * 4);
#pragma unroll
  for (int h = 0; h < 2; ++h) {
#pragma unroll
    for (int r = 0; r < 16; ++r) {
      const int acc = h == 0 ? acc0[r] : acc1[r];
      const float y = (float)acc * scale;
      const float z = __builtin_amdgcn_fmed3f(y, -60.0f, 60.0f);
      const float e = __builtin_amdgcn_exp2f(z * -1.44269504088896340736f);
      const float v = y * __builtin_amdgcn_rcpf(1.0f + e);
      __builtin_amdgcn_raw_buffer_store_b32(__builtin_bit_cast(unsigned, v), ors, voff, (h * 32 + (r & 3) + 8 * (r >> 2)) * NDIM * 4, OUT_AUX);
    }
  }
}

}

extern "C" void kernel_launch(int rank, int nranks, ncclComm_t comm, hipStream_t stream,
                              void* const* d_in, const int* in_sizes, int n_in,
                              void* d_out, int out_size, const int* in_dtypes, int out_dtype,
                              void* const* d_out_all, void* const* d_ws_all, size_t ws_bytes) {
  (void)comm; (void)in_sizes; (void)in_dtypes; (void)out_dtype; (void)d_out_all;
  if (nranks != NR || n_in != 4 || out_size != MS * NDIM || ws_bytes < WS_NEED || rank < 0 || rank >= NR) {
    fprintf(stderr, "kernel_launch: unexpected configuration\n");
    return;
  }
  static bool attr_set[NR] = {};
  if (!attr_set[rank]) {
    HIP_CHECK(hipFuncSetAttribute((const void*)gemm_ag_kernel, hipFuncAttributeMaxDynamicSharedMemorySize, LDS_BYTES));
    attr_set[rank] = true;
  }
  Params p;
  p.x = (const char*)d_in[0];
  p.w = (const char*)d_in[1];
  p.sx = (const float*)d_in[2];
  p.sw = (const float*)d_in[3];
  p.out = (float*)d_out;
  for (int i = 0; i < NR; ++i) p.ws[i] = (char*)d_ws_all[i];
  p.myws = p.ws[rank];
  p.rank = rank;
  p.pad = 0;
  hipLaunchKernelGGL(gemm_ag_kernel, dim3(GRID), dim3(THREADS), LDS_BYTES, stream, p);
  HIP_CHECK(hipGetLastError());
}
```

```cpp
#include <hip/hip_runtime.h>
#include <rccl/rccl.h>
#include <cstdio>

#define HIP_CHECK(expr) do { hipError_t _e = (expr); if (_e != hipSuccess) { \
    fprintf(stderr, "HIP error %s at %s:%d\n", hipGetErrorString(_e), __FILE__, __LINE__); \
    return; } } while (0)

#define GAS __attribute__((address_space(1)))
#define LAS __attribute__((address_space(3)))
typedef unsigned u32x4 __attribute__((ext_vector_type(4)));
typedef int i32x4 __attribute__((ext_vector_type(4)));
typedef int i32x16 __attribute__((ext_vector_type(16)));
typedef unsigned long long u64;

#define DEPTH 6
#define PACE16 448
#define STAG 0
#define XPACE 112
#define PFD 4
#define XDELAY 90
#define GODELAY 430
#define PMARGIN 20
#define RAMP 10
#define DELTA 80
#define PDEPTH 6
#define OUT_AUX 17
#define XSLEEP 8
#define LPRIO 3

namespace {

constexpr int NR = 8;
constexpr int NDIM = 4096;
constexpr int KS = 512;
constexpr int MS = 512;
constexpr int NSLAB = NDIM / 32;
constexpr int NKS = KS / 32;
constexpr int NUNIT = NR * NKS;
constexpr int N_WST = 64, N_STAGE = 128;
constexpr int GRID = N_STAGE + NSLAB;
constexpr int THREADS = 576;

constexpr size_t OFF_CNT = 0;
constexpr size_t OFF_TMO = OFF_CNT + GRID * 128;
constexpr size_t OFF_WCNT = OFF_TMO + 128;
constexpr size_t OFF_FC = OFF_WCNT + 8 * 128;
constexpr size_t OFF_GO = OFF_FC + 8 * 128;
constexpr size_t OFF_XFLAG = OFF_GO + 8 * 128;
constexpr size_t OFF_STAMP = 64 << 10;
constexpr size_t STAGE_BYTES = 2u << 20;
constexpr size_t OFF_X = 1u << 20;
constexpr size_t OFF_W = OFF_X + 2 * STAGE_BYTES;
constexpr size_t WS_NEED = OFF_W + 2 * STAGE_BYTES;
static_assert(OFF_XFLAG + 8 * 256 <= OFF_STAMP, "layout");

constexpr int LDS_B = NUNIT * 1024;
constexpr int LDS_CTL = LDS_B;
constexpr int LDS_BYTES = LDS_B + 64;

constexpr u64 DEADLINE_TICKS = 50000000ull;

struct Params {
  const char* x;
  const char* w;
  const float* sx;
  const float* sw;
  float* out;
  char* myws;
  char* ws[NR];
  int rank;
  int pad;
};

__device__ __forceinline__ u64 rt() { return __builtin_amdgcn_s_memrealtime(); }
template <int N> __device__ __forceinline__ void wait_vm() { asm volatile("s_waitcnt vmcnt(%0)" :: "i"(N) : "memory"); }
__device__ __forceinline__ void store_wt(void* p, u32x4 v) {
  asm volatile("global_store_dwordx4 %0, %1, off sc0 sc1\n\ts_nop 1" :: "v"(p), "v"(v) : "memory");
}
__device__ __forceinline__ void lds_write(unsigned addr, unsigned v) {
  asm volatile("ds_write_b32 %0, %1" :: "v"(addr), "v"(v) : "memory");
}
template <int D> __device__ __forceinline__ void drain(unsigned addr) {
  if constexpr (D > 0) {
    wait_vm<D>();
    lds_write(addr, NUNIT - D);
    drain<D - 1>(addr);
  }
}
__device__ __forceinline__ unsigned lds_read(unsigned addr) {
  unsigned v;
  asm volatile("ds_read_b32 %0, %1\n\ts_waitcnt lgkmcnt(0)" : "=v"(v) : "v"(addr) : "memory");
  return v;
}
__device__ __forceinline__ unsigned ld_sys(const void* p) {
  return __hip_atomic_load((const GAS unsigned*)p, __ATOMIC_RELAXED, __HIP_MEMORY_SCOPE_SYSTEM);
}
__device__ __forceinline__ unsigned ld_agent(const void* p) {
  return __hip_atomic_load((const GAS unsigned*)p, __ATOMIC_RELAXED, __HIP_MEMORY_SCOPE_AGENT);
}
__device__ __forceinline__ void st_sys(void* p, unsigned v) {
  __hip_atomic_store((GAS unsigned*)p, v, __ATOMIC_RELAXED, __HIP_MEMORY_SCOPE_SYSTEM);
}
__device__ __forceinline__ void st_agent(void* p, unsigned v) {
  __hip_atomic_store((GAS unsigned*)p, v, __ATOMIC_RELAXED, __HIP_MEMORY_SCOPE_AGENT);
}
__device__ __forceinline__ unsigned perm(unsigned hi, unsigned lo, unsigned sel) { return __builtin_amdgcn_perm(hi, lo, sel); }
#define SGRP(slab) (((slab) >> 3) & 1)
__device__ __forceinline__ unsigned sched(int t, int grp) {
  constexpr unsigned T = NUNIT * PACE16;
  constexpr unsigned S = STAG * 16;
  if (STAG == 0) return (unsigned)(t * PACE16) >> 4;
  constexpr unsigned P0 = (T - S) / NUNIT;
  constexpr double r1 = 2.0 / PACE16 - 1.0 / P0;
  constexpr unsigned P1 = (unsigned)(1.0 / r1 + 0.5);
  constexpr int N1g = (int)((T - S) / P1);
  if (grp == 0) return (unsigned)(t * P0) >> 4;
  return t < N1g ? (unsigned)(t * P1) >> 4 : (unsigned)(N1g * P1 + (t - N1g) * (PACE16 / 2)) >> 4;
}
__device__ __forceinline__ void wait_until(unsigned t) {
  while ((int)((unsigned)rt() - t) < 0) __builtin_amdgcn_s_sleep(1);
}

#define STAMP(i) do { } while (0)

template <int SLEEP>
__device__ __forceinline__ bool wait_flags(const char* base, int n, int lane, int stride, unsigned target, unsigned call, char* tmo) {
  const u64 t0 = rt();
  const char* q = base + (lane < n ? lane : 0) * stride;
  for (unsigned it = 1;; ++it) {
    const unsigned v = ld_sys(q);
    if (__all(v >= target)) return true;
    if ((it & 63) == 0 && (ld_agent(tmo) == call || rt() - t0 > DEADLINE_TICKS)) { st_agent(tmo, call); return false; }
    __builtin_amdgcn_s_sleep(SLEEP);
  }
}

__device__ __forceinline__ const char* frag_src(u64 tab, int t, int lane) {
  const unsigned lo = __builtin_amdgcn_readlane((unsigned)tab, t & 7), hi = __builtin_amdgcn_readlane((unsigned)(tab >> 32), t & 7);
  return (const char*)(((u64)hi << 32) | lo) + (t >> 3) * 1024 + lane * 16;
}

__global__ void __launch_bounds__(THREADS) gemm_ag_kernel(Params p) {
  extern __shared__ __attribute__((aligned(16))) char lds[];
  const int tid = threadIdx.x;
  const int wave = __builtin_amdgcn_readfirstlane(tid >> 6);
  const int lane = tid & 63;
  const int bid = blockIdx.x;
  char* const myws = p.myws;
  char* const tmo = myws + OFF_TMO;
  const unsigned ctl = (unsigned)(size_t)(LAS char*)lds + LDS_CTL;

  char* const cntp = myws + OFF_CNT + bid * 128;
  unsigned cnt_v = ld_agent(cntp);
  if (wave == 0) STAMP(0);
#define GET_CALL() ({ asm volatile("" : "+v"(cnt_v) :: "memory"); (unsigned)__builtin_amdgcn_readfirstlane(cnt_v) + 1; })
  const unsigned t_start = (unsigned)rt();

  if (bid < N_WST) {
    const int G = bid >> 1, kh = bid & 1;
    const int pslab = bid + (wave & 1) * 64, pj0 = (pslab + (pslab >> 3)) & (NR - 1);
    u64 tab = (u64)p.ws[wave == 2 ? (lane & (NR - 1)) : ((pj0 + lane) & (NR - 1))];
    unsigned call;
    if (wave < 8) {
      const int ks = kh * 8 + wave;
      unsigned r[16];
      u32x4 v[4];
      const char* src = p.w + (size_t)(ks * 32 + (lane >> 5) * 16) * NDIM + G * 128 + (lane & 31) * 4;
#pragma unroll
      for (int i = 0; i < 16; ++i) r[i] = *(const GAS unsigned*)(src + (size_t)i * NDIM);
      call = GET_CALL();
#pragma unroll
      for (int g = 0; g < 4; ++g) {
        const unsigned lo01 = perm(r[4 * g + 1], r[4 * g], 0x05010400u), hi01 = perm(r[4 * g + 1], r[4 * g], 0x07030602u);
        const unsigned lo23 = perm(r[4 * g + 3], r[4 * g + 2], 0x05010400u), hi23 = perm(r[4 * g + 3], r[4 * g + 2], 0x07030602u);
        v[0][g] = perm(lo23, lo01, 0x05040100u);
        v[1][g] = perm(lo23, lo01, 0x07060302u);
        v[2][g] = perm(hi23, hi01, 0x05040100u);
        v[3][g] = perm(hi23, hi01, 0x07060302u);
      }
      const int slab = G * 4 + ((lane >> 3) & 3);
      char* dst = myws + OFF_W + (size_t)(call & 1) * STAGE_BYTES + (size_t)(slab * NKS + ks) * 1024 + ((lane & 7) + (lane >> 5) * 32) * 16;
#pragma unroll
      for (int q = 0; q < 4; ++q) store_wt(dst + q * 128, v[q]);
      wait_vm<0>();
    } else {
      call = GET_CALL();
    }
    asm volatile("s_barrier" ::: "memory");
    if (tid == 0) {
      __hip_atomic_fetch_add((GAS unsigned*)(myws + OFF_WCNT + (bid & 7) * 128), 1u, __ATOMIC_RELAXED, __HIP_MEMORY_SCOPE_AGENT);
      st_agent(cntp, call);
    }
    if (wave == 0) STAMP(1);
    if (bid == 0 && wave == 2) {
      const bool ok = wait_flags<1>(myws + OFF_WCNT, 8, lane, 128, call * (N_WST / 8), call, tmo);
      wait_until(t_start + GODELAY);
      if (ok) st_sys((char*)tab + OFF_GO + (lane >> 3) * 128 + p.rank * 4, call);
      STAMP(2);
    }
    if (wave < 2) {
      tab += OFF_W + (size_t)(call & 1) * STAGE_BYTES + (size_t)pslab * NKS * 1024;
      if (!wait_flags<1>(myws + OFF_GO + (bid & 7) * 128, 8, lane, 4, call, call, tmo)) return;
      const unsigned t0 = (unsigned)rt();
      if (wave == 0) STAMP(3);
      if (!wait_flags<4>(myws + OFF_FC + (bid & 7) * 128, 1, lane, 4, call * (NSLAB / 8), call, tmo)) return;
      const unsigned tn = (unsigned)rt() - t0 + PMARGIN;
      if (wave == 0) STAMP(5);
      for (int t = 0; t < NUNIT; ++t) {
        const unsigned ts = sched(t, SGRP(pslab));
        if (ts < tn || ((pj0 + t) & (NR - 1)) == p.rank) continue;
        const char* s = frag_src(tab, t, lane);
        wait_until(t0 + ts);
        __builtin_amdgcn_global_load_lds((const GAS unsigned*)s, (LAS unsigned*)(lds + ((wave * 16 + (t & 15)) << 10)), 16, 0, 0);
        wait_vm<PDEPTH>();
      }
      wait_vm<0>();
      if (wave == 0) STAMP(4);
    }
    return;
  }
  if (bid < N_STAGE) {
    const int idx = bid - N_WST;
    const int dst_rank = (p.rank + (idx >> 3)) & (NR - 1), part = idx & 7;
    char* const dws = p.ws[dst_rank];
    unsigned call = 0;
    if (wave < 8) {
      const int rtile = part * 2 + (wave & 1), ks0 = (wave >> 1) * 4;
      u32x4 v[4];
      const char* src = p.x + (size_t)(dst_rank * MS + rtile * 32 + (lane & 31)) * KS + ks0 * 32 + (lane >> 5) * 16;
#pragma unroll
      for (int i = 0; i < 4; ++i) v[i] = *(const GAS u32x4*)(src + i * 32);
      call = GET_CALL();
      char* xdst = dws + OFF_X + (size_t)(call & 1) * STAGE_BYTES + (size_t)((p.rank * NKS + ks0) * 16 + rtile) * 1024 + lane * 16;
#pragma unroll
      for (int i = 0; i < 4; ++i) {
        wait_until(t_start + XDELAY + ((wave * XPACE) >> 3) + i * XPACE);
        store_wt(xdst + i * 16384, v[i]);
      }
    }
    wait_vm<0>();
    asm volatile("s_barrier" ::: "memory");
    if (tid < 8) st_sys(dws + OFF_XFLAG + tid * 256 + (p.rank * 8 + part) * 4, call);
    if (tid == 0) st_agent(cntp, call);
    if (wave == 0) STAMP(2);
    return;
  }

  const int slab = bid - N_STAGE;
  const int j0 = (slab + (slab >> 3)) & (NR - 1);
  u64 tab = 0;
  if (wave == 8) tab = (u64)p.ws[(j0 + lane) & (NR - 1)];
  const unsigned call = GET_CALL();
  if (tid == 0) {
    lds_write(ctl + 4, 0); lds_write(ctl + 8, 0); lds_write(ctl + 12, 0); lds_write(ctl + 16, 0); lds_write(ctl + 20, 0);
    asm volatile("s_waitcnt lgkmcnt(0)" ::: "memory");
  }
  asm volatile("s_barrier" ::: "memory");
  if (tid == 0) st_agent(cntp, call);

  if (wave == 8) {
    tab += OFF_W + (size_t)(call & 1) * STAGE_BYTES + (size_t)slab * NKS * 1024;
    const bool ok = wait_flags<1>(myws + OFF_GO + (bid & 7) * 128, 8, lane, 4, call, call, tmo);
    const unsigned t0 = (unsigned)rt();
    STAMP(1);
    if (!ok) {
      lds_write(ctl + 12, 1);
      lds_write(ctl + 16, 1);
      lds_write(ctl + 4, NUNIT);
      return;
    }
    lds_write(ctl + 16, 1);
    unsigned via_l2 = 0, delta = 0;
    const unsigned te = sched(NUNIT - 1, SGRP(slab));
    for (int t = 0; t < NUNIT; ++t) {
      const char* s = frag_src(tab, t, lane);
      const unsigned ts = sched(t, SGRP(slab));
      if (!via_l2) {
        wait_until(t0 + ts);
        via_l2 = __builtin_amdgcn_readfirstlane(lds_read(ctl + 20));
      }
      if (via_l2) {
        wait_until(t0 + (ts + delta < te ? ts + delta : te));
        delta = delta + RAMP < DELTA ? delta + RAMP : DELTA;
        __builtin_amdgcn_s_setprio(LPRIO);
        __builtin_amdgcn_global_load_lds((const GAS unsigned*)s, (LAS unsigned*)(lds + t * 1024), 16, 0, 0);
        __builtin_amdgcn_s_setprio(0);
      } else {
        __builtin_amdgcn_global_load_lds((const GAS unsigned*)s, (LAS unsigned*)(lds + t * 1024), 16, 0, 17);
      }
      wait_vm<DEPTH>();
      if (t + 1 - DEPTH > 0) lds_write(ctl + 4, t + 1 - DEPTH);
    }
    drain<DEPTH - 1>(ctl + 4);
    wait_vm<0>();
    lds_write(ctl + 4, NUNIT);
    STAMP(2);
    return;
  }

  float scale0 = p.sx[0] * p.sw[0];
  unsigned out_lo = (unsigned)(size_t)p.out, out_hi = (unsigned)((size_t)p.out >> 32);
  asm volatile("" : "+v"(scale0), "+v"(out_lo), "+v"(out_hi));
  if (wave == 0) {
    const bool ok = wait_flags<XSLEEP>(myws + OFF_XFLAG + (bid & 7) * 256, 64, lane, 4, call, call, tmo);
    while (lds_read(ctl + 16) == 0) __builtin_amdgcn_s_sleep(1);
    STAMP(9);
    __builtin_amdgcn_fence(__ATOMIC_ACQUIRE, "");
    asm volatile("s_waitcnt vmcnt(0)" ::: "memory");
    if (lane == 0) __hip_atomic_fetch_add((GAS unsigned*)(myws + OFF_FC + (bid & 7) * 128), 1u, __ATOMIC_RELAXED, __HIP_MEMORY_SCOPE_AGENT);
    STAMP(3);
    const bool ok2 = wait_flags<2>(myws + OFF_FC + (bid & 7) * 128, 1, lane, 4, call * (NSLAB / 8), call, tmo);
    if (!ok || !ok2) lds_write(ctl + 12, 1);
    lds_write(ctl + 20, 1);
    STAMP(10);
    if (ld_agent(tmo) == call) lds_write(ctl + 12, 1);
  } else {
    while (lds_read(ctl + 20) == 0) __builtin_amdgcn_s_sleep(4);
  }
  asm volatile("" ::: "memory");

  const char* xa = myws + OFF_X + (size_t)(call & 1) * STAGE_BYTES + wave * 2048 + lane * 16;
  auto a_addr = [&](int t) { return xa + (size_t)((((j0 + t) & (NR - 1)) * NKS + (t >> 3)) << 14); };
  constexpr int PF = PFD;
  static_assert(NUNIT % PF == 0, "prefetch depth");
  i32x4 a[PF][2];
#pragma unroll
  for (int i = 0; i < PF; ++i) {
    const char* q = a_addr(i);
    a[i][0] = *(const GAS i32x4*)q;
    a[i][1] = *(const GAS i32x4*)(q + 1024);
  }
  i32x16 acc0 = {}, acc1 = {};
  unsigned landed = 0;
  auto step = [&](int t, int i, bool prefetch) {
    while (landed <= (unsigned)t) {
      landed = __builtin_amdgcn_readfirstlane(lds_read(ctl + 4));
      if (landed <= (unsigned)t) __builtin_amdgcn_s_sleep(1);
    }
    const i32x4 b = *(const LAS i32x4*)(lds + t * 1024 + lane * 16);
    acc0 = __builtin_amdgcn_mfma_i32_32x32x32_i8(a[i][0], b, acc0, 0, 0, 0);
    acc1 = __builtin_amdgcn_mfma_i32_32x32x32_i8(a[i][1], b, acc1, 0, 0, 0);
    if (prefetch) {
      const char* q = a_addr(t + PF);
      a[i][0] = *(const GAS i32x4*)q;
      a[i][1] = *(const GAS i32x4*)(q + 1024);
    }
  };
  for (int t0 = 0; t0 < NUNIT - PF; t0 += PF) {
#pragma unroll
    for (int i = 0; i < PF; ++i) step(t0 + i, i, true);
  }
#pragma unroll
  for (int i = 0; i < PF; ++i) step(NUNIT - PF + i, i, false);
  if (wave == 0) STAMP(4);
  STAMP(16 + wave);

  const float scale = lds_read(ctl + 12) != 0 ? __builtin_nanf("") : scale0;
  const int c31 = lane & 31;
  float* outp = (float*)(((size_t)(unsigned)__builtin_amdgcn_readfirstlane(out_hi) << 32) | (unsigned)__builtin_amdgcn_readfirstlane(out_lo));
  const __amdgpu_buffer_rsrc_t ors = __builtin_amdgcn_make_buffer_rsrc(outp, 0, MS * NDIM * 4, 0x00020000);
  const unsigned voff = (unsigned)(((wave * 64 + (lane >> 5) * 4) * NDIM + slab * 32 + (c31 & 7) * 4 + (c31 >> 3)) * 4);
#pragma unroll
  for (int h = 0; h < 2; ++h) {
#pragma unroll
    for (int r = 0; r < 16; ++r) {
      const int acc = h == 0 ? acc0[r] : acc1[r];
      const float y = (float)acc * scale;
      const float z = __builtin_amdgcn_fmed3f(y, -60.0f, 60.0f);
      const float e = __builtin_amdgcn_exp2f(z * -1.44269504088896340736f);
      const float v = y * __builtin_amdgcn_rcpf(1.0f + e);
      __builtin_amdgcn_raw_buffer_store_b32(__builtin_bit_cast(unsigned, v), ors, voff, (h * 32 + (r & 3) + 8 * (r >> 2)) * NDIM * 4, OUT_AUX);
    }
  }
}

}

extern "C" void kernel_launch(int rank, int nranks, ncclComm_t comm, hipStream_t stream,
                              void* const* d_in, const int* in_sizes, int n_in,
                              void* d_out, int out_size, const int* in_dtypes, int out_dtype,
                              void* const* d_out_all, void* const* d_ws_all, size_t ws_bytes) {
  (void)comm; (void)in_sizes; (void)in_dtypes; (void)out_dtype; (void)d_out_all;
  if (nranks != NR || n_in != 4 || out_size != MS * NDIM || ws_bytes < WS_NEED || rank < 0 || rank >= NR) {
    fprintf(stderr, "kernel_launch: unexpected configuration\n");
    return;
  }
  static bool attr_set[NR] = {};
  if (!attr_set[rank]) {
    HIP_CHECK(hipFuncSetAttribute((const void*)gemm_ag_kernel, hipFuncAttributeMaxDynamicSharedMemorySize, LDS_BYTES));
    attr_set[rank] = true;
  }
  Params p;
  p.x = (const char*)d_in[0];
  p.w = (const char*)d_in[1];
  p.sx = (const float*)d_in[2];
  p.sw = (const float*)d_in[3];
  p.out = (float*)d_out;
  for (int i = 0; i < NR; ++i) p.ws[i] = (char*)d_ws_all[i];
  p.myws = p.ws[rank];
  p.rank = rank;
  p.pad = 0;
  hipLaunchKernelGGL(gemm_ag_kernel, dim3(GRID), dim3(THREADS), LDS_BYTES, stream, p);
  HIP_CHECK(hipGetLastError());
}
```

```cpp
#include <hip/hip_runtime.h>
#include <rccl/rccl.h>
#include <cstdio>

#define HIP_CHECK(expr) do { hipError_t _e = (expr); if (_e != hipSuccess) { \
    fprintf(stderr, "HIP error %s at %s:%d\n", hipGetErrorString(_e), __FILE__, __LINE__); \
    return; } } while (0)

#define GAS __attribute__((address_space(1)))
#define LAS __attribute__((address_space(3)))
typedef unsigned u32x4 __attribute__((ext_vector_type(4)));
typedef int i32x4 __attribute__((ext_vector_type(4)));
typedef int i32x16 __attribute__((ext_vector_type(16)));
typedef float f32x2 __attribute__((ext_vector_type(2)));
typedef unsigned long long u64;

#define DEPTH 6
#define PACE16 448
#define STAG 0
#define XPACE 112
#define PFD 4
#define XDELAY 90
#define GODELAY 430
#define LDEPTH 4
#define PSKIP 3
#define PAUX 0
#define LAUX 0
#define CPACE16 0
#define RAMP 10
#define DELTA 80
#define PDEPTH 6
#define OUT_AUX 17
#define XSLEEP 8
#define LPRIO 3

namespace {

constexpr int NR = 8;
constexpr int NDIM = 4096;
constexpr int KS = 512;
constexpr int MS = 512;
constexpr int NSLAB = NDIM / 32;
constexpr int NKS = KS / 32;
constexpr int NUNIT = NR * NKS;
constexpr int N_WST = 64, N_STAGE = 128;
constexpr int GRID = N_STAGE + NSLAB;
constexpr int THREADS = 576;

constexpr size_t OFF_CNT = 0;
constexpr size_t OFF_TMO = OFF_CNT + GRID * 128;
constexpr size_t OFF_WCNT = OFF_TMO + 128;
constexpr size_t OFF_FC = OFF_WCNT + 8 * 128;
constexpr size_t OFF_GO = OFF_FC + 8 * 128;
constexpr size_t OFF_XFLAG = OFF_GO + 8 * 128;
constexpr size_t OFF_SW = 48 << 10;
constexpr size_t OFF_STAMP = 64 << 10;
constexpr size_t STAGE_BYTES = 2u << 20;
constexpr size_t OFF_X = 1u << 20;
constexpr size_t OFF_W = OFF_X + 2 * STAGE_BYTES;
constexpr size_t WS_NEED = OFF_W + 2 * STAGE_BYTES;
static_assert(OFF_XFLAG + 8 * 256 <= OFF_SW && OFF_SW + NSLAB * 128 <= OFF_STAMP, "layout");

constexpr int LDS_B = NUNIT * 1024;
constexpr int LDS_CTL = LDS_B;
constexpr int LDS_BYTES = LDS_B + 64;

constexpr u64 DEADLINE_TICKS = 50000000ull;

struct Params {
  const char* x;
  const char* w;
  const float* sx;
  const float* sw;
  float* out;
  char* myws;
  char* ws[NR];
  int rank;
  int pad;
};

__device__ __forceinline__ u64 rt() { return __builtin_amdgcn_s_memrealtime(); }
template <int N> __device__ __forceinline__ void wait_vm() { asm volatile("s_waitcnt vmcnt(%0)" :: "i"(N) : "memory"); }
__device__ __forceinline__ void store_wt(void* p, u32x4 v) {
  asm volatile("global_store_dwordx4 %0, %1, off sc0 sc1\n\ts_nop 1" :: "v"(p), "v"(v) : "memory");
}
__device__ __forceinline__ void lds_write(unsigned addr, unsigned v) {
  asm volatile("ds_write_b32 %0, %1" :: "v"(addr), "v"(v) : "memory");
}
template <int D> __device__ __forceinline__ void drain(unsigned addr) {
  if constexpr (D > 0) {
    wait_vm<D>();
    lds_write(addr, NUNIT - D);
    drain<D - 1>(addr);
  }
}
__device__ __forceinline__ unsigned lds_read(unsigned addr) {
  unsigned v;
  asm volatile("ds_read_b32 %0, %1\n\ts_waitcnt lgkmcnt(0)" : "=v"(v) : "v"(addr) : "memory");
  return v;
}
__device__ __forceinline__ unsigned ld_sys(const void* p) {
  return __hip_atomic_load((const GAS unsigned*)p, __ATOMIC_RELAXED, __HIP_MEMORY_SCOPE_SYSTEM);
}
__device__ __forceinline__ unsigned ld_agent(const void* p) {
  return __hip_atomic_load((const GAS unsigned*)p, __ATOMIC_RELAXED, __HIP_MEMORY_SCOPE_AGENT);
}
__device__ __forceinline__ void st_sys(void* p, unsigned v) {
  __hip_atomic_store((GAS unsigned*)p, v, __ATOMIC_RELAXED, __HIP_MEMORY_SCOPE_SYSTEM);
}
__device__ __forceinline__ void st_agent(void* p, unsigned v) {
  __hip_atomic_store((GAS unsigned*)p, v, __ATOMIC_RELAXED, __HIP_MEMORY_SCOPE_AGENT);
}
__device__ __forceinline__ unsigned perm(unsigned hi, unsigned lo, unsigned sel) { return __builtin_amdgcn_perm(hi, lo, sel); }
#define SGRP(slab) (((slab) >> 3) & 1)
__device__ __forceinline__ unsigned sched(int t, int grp) {
  constexpr unsigned T = NUNIT * PACE16;
  constexpr unsigned S = STAG * 16;
  if (STAG == 0) return (unsigned)(t * PACE16) >> 4;
  constexpr unsigned P0 = (T - S) / NUNIT;
  constexpr double r1 = 2.0 / PACE16 - 1.0 / P0;
  constexpr unsigned P1 = (unsigned)(1.0 / r1 + 0.5);
  constexpr int N1g = (int)((T - S) / P1);
  if (grp == 0) return (unsigned)(t * P0) >> 4;
  return t < N1g ? (unsigned)(t * P1) >> 4 : (unsigned)(N1g * P1 + (t - N1g) * (PACE16 / 2)) >> 4;
}
__device__ __forceinline__ void wait_until(unsigned t) {
  while ((int)((unsigned)rt() - t) < 0) __builtin_amdgcn_s_sleep(1);
}

#define STAMP(i) do { } while (0)

template <int SLEEP>
__device__ __forceinline__ bool wait_flags(const char* base, int n, int lane, int stride, unsigned target, unsigned call, char* tmo) {
  const u64 t0 = rt();
  const char* q = base + (lane < n ? lane : 0) * stride;
  for (unsigned it = 1;; ++it) {
    const unsigned v = ld_sys(q);
    if (__all(v >= target)) return true;
    if ((it & 63) == 0 && (ld_agent(tmo) == call || rt() - t0 > DEADLINE_TICKS)) { st_agent(tmo, call); return false; }
    __builtin_amdgcn_s_sleep(SLEEP);
  }
}

__device__ __forceinline__ const char* frag_src(u64 tab, int t, int lane) {
  const unsigned lo = __builtin_amdgcn_readlane((unsigned)tab, t & 7), hi = __builtin_amdgcn_readlane((unsigned)(tab >> 32), t & 7);
  return (const char*)(((u64)hi << 32) | lo) + (t >> 3) * 1024 + lane * 16;
}

__global__ void __launch_bounds__(THREADS) gemm_ag_kernel(Params p) {
  extern __shared__ __attribute__((aligned(16))) char lds[];
  const int tid = threadIdx.x;
  const int wave = __builtin_amdgcn_readfirstlane(tid >> 6);
  const int lane = tid & 63;
  const int bid = blockIdx.x;
  char* const myws = p.myws;
  char* const tmo = myws + OFF_TMO;
  const unsigned ctl = (unsigned)(size_t)(LAS char*)lds + LDS_CTL;

  char* const cntp = myws + OFF_CNT + bid * 128;
  unsigned cnt_v = ld_agent(cntp);
  if (wave == 0) STAMP(0);
#define GET_CALL() ({ asm volatile("" : "+v"(cnt_v) :: "memory"); (unsigned)__builtin_amdgcn_readfirstlane(cnt_v) + 1; })
  const unsigned t_start = (unsigned)rt();

  if (bid < N_WST) {
    const int G = bid >> 1, kh = bid & 1;
    const int pslab = bid + (wave & 1) * 64, pj0 = (pslab + (pslab >> 3)) & (NR - 1);
    u64 tab = (u64)p.ws[wave == 2 ? (lane & (NR - 1)) : ((pj0 + lane) & (NR - 1))];
    unsigned call;
    if (wave < 8) {
      const int ks = kh * 8 + wave;
      unsigned r[16];
      u32x4 v[4];
      const char* src = p.w + (size_t)(ks * 32 + (lane >> 5) * 16) * NDIM + G * 128 + (lane & 31) * 4;
#pragma unroll
      for (int i = 0; i < 16; ++i) r[i] = *(const GAS unsigned*)(src + (size_t)i * NDIM);
      call = GET_CALL();
#pragma unroll
      for (int g = 0; g < 4; ++g) {
        const unsigned lo01 = perm(r[4 * g + 1], r[4 * g], 0x05010400u), hi01 = perm(r[4 * g + 1], r[4 * g], 0x07030602u);
        const unsigned lo23 = perm(r[4 * g + 3], r[4 * g + 2], 0x05010400u), hi23 = perm(r[4 * g + 3], r[4 * g + 2], 0x07030602u);
        v[0][g] = perm(lo23, lo01, 0x05040100u);
        v[1][g] = perm(lo23, lo01, 0x07060302u);
        v[2][g] = perm(hi23, hi01, 0x05040100u);
        v[3][g] = perm(hi23, hi01, 0x07060302u);
      }
      const int slab = G * 4 + ((lane >> 3) & 3);
      char* dst = myws + OFF_W + (size_t)(call & 1) * STAGE_BYTES + (size_t)(slab * NKS + ks) * 1024 + ((lane & 7) + (lane >> 5) * 32) * 16;
#pragma unroll
      for (int q = 0; q < 4; ++q) store_wt(dst + q * 128, v[q]);
      wait_vm<0>();
    } else {
      call = GET_CALL();
    }
    asm volatile("s_barrier" ::: "memory");
    if (tid == 0) {
      __hip_atomic_fetch_add((GAS unsigned*)(myws + OFF_WCNT + (bid & 7) * 128), 1u, __ATOMIC_RELAXED, __HIP_MEMORY_SCOPE_AGENT);
      st_agent(cntp, call);
    }
    if (wave == 0) STAMP(1);
    if (bid == 0 && wave == 2) {
      const bool ok = wait_flags<1>(myws + OFF_WCNT, 8, lane, 128, call * (N_WST / 8), call, tmo);
      wait_until(t_start + GODELAY);
      if (ok) st_sys((char*)tab + OFF_GO + (lane >> 3) * 128 + p.rank * 4, call);
      STAMP(2);
    }
    if (wave < 2) {
      tab += OFF_W + (size_t)(call & 1) * STAGE_BYTES + (size_t)pslab * NKS * 1024;
      if (!wait_flags<1>(myws + OFF_GO + (bid & 7) * 128, 8, lane, 4, call, call, tmo)) return;
      const unsigned t0 = (unsigned)rt();
      if (wave == 0) STAMP(3);
      u64 m;
      {
        const u64 tw0 = rt();
        for (unsigned it = 1;; ++it) {
          m = __hip_atomic_load((const GAS u64*)(myws + OFF_SW + pslab * 128), __ATOMIC_RELAXED, __HIP_MEMORY_SCOPE_AGENT);
          if (__builtin_amdgcn_readfirstlane((unsigned)(m >> 32)) == call) break;
          if ((it & 63) == 0 && (ld_agent(tmo) == call || rt() - tw0 > DEADLINE_TICKS)) { st_agent(tmo, call); return; }
          __builtin_amdgcn_s_sleep(2);
        }
      }
      const unsigned mw = __builtin_amdgcn_readfirstlane((unsigned)m);
      const unsigned tp = t0 + (mw & 0xffffu);
      if (wave == 0) STAMP(5);
      for (int t = (int)(mw >> 16) + PSKIP; t < NUNIT; ++t) {
        if (((pj0 + t) & (NR - 1)) == p.rank) continue;
        const char* s = frag_src(tab, t, lane);
        wait_until(tp + sched(t, SGRP(pslab)));
        __builtin_amdgcn_global_load_lds((const GAS unsigned*)s, (LAS unsigned*)(lds + ((wave * 16 + (t & 15)) << 10)), 16, 0, PAUX);
        wait_vm<PDEPTH>();
      }
      wait_vm<0>();
      if (wave == 0) STAMP(4);
    }
    return;
  }
  if (bid < N_STAGE) {
    const int idx = bid - N_WST;
    const int dst_rank = (p.rank + (idx >> 3)) & (NR - 1), part = idx & 7;
    char* const dws = p.ws[dst_rank];
    unsigned call = 0;
    if (wave < 8) {
      const int rtile = part * 2 + (wave & 1), ks0 = (wave >> 1) * 4;
      u32x4 v[4];
      const char* src = p.x + (size_t)(dst_rank * MS + rtile * 32 + (lane & 31)) * KS + ks0 * 32 + (lane >> 5) * 16;
#pragma unroll
      for (int i = 0; i < 4; ++i) v[i] = *(const GAS u32x4*)(src + i * 32);
      call = GET_CALL();
      char* xdst = dws + OFF_X + (size_t)(call & 1) * STAGE_BYTES + (size_t)((p.rank * NKS + ks0) * 16 + rtile) * 1024 + lane * 16;
#pragma unroll
      for (int i = 0; i < 4; ++i) {
        wait_until(t_start + XDELAY + ((wave * XPACE) >> 3) + i * XPACE);
        store_wt(xdst + i * 16384, v[i]);
      }
    }
    wait_vm<0>();
    asm volatile("s_barrier" ::: "memory");
    if (tid < 8) st_sys(dws + OFF_XFLAG + tid * 256 + (p.rank * 8 + part) * 4, call);
    if (tid == 0) st_agent(cntp, call);
    if (wave == 0) STAMP(2);
    return;
  }

  const int slab = bid - N_STAGE;
  const int j0 = (slab + (slab >> 3)) & (NR - 1);
  u64 tab = 0;
  if (wave == 8) tab = (u64)p.ws[(j0 + lane) & (NR - 1)];
  const unsigned call = GET_CALL();
  if (tid == 0) {
    lds_write(ctl + 4, 0); lds_write(ctl + 8, 0); lds_write(ctl + 12, 0); lds_write(ctl + 16, 0); lds_write(ctl + 20, 0);
    asm volatile("s_waitcnt lgkmcnt(0)" ::: "memory");
  }
  asm volatile("s_barrier" ::: "memory");
  if (tid == 0) st_agent(cntp, call);

  if (wave == 8) {
    tab += OFF_W + (size_t)(call & 1) * STAGE_BYTES + (size_t)slab * NKS * 1024;
    const bool ok = wait_flags<1>(myws + OFF_GO + (bid & 7) * 128, 8, lane, 4, call, call, tmo);
    const unsigned t0 = (unsigned)rt();
    STAMP(1);
    if (!ok) {
      lds_write(ctl + 12, 1);
      lds_write(ctl + 16, 1);
      lds_write(ctl + 4, NUNIT);
      return;
    }
    lds_write(ctl + 16, 1);
    unsigned via_l2 = 0, delta = 0, lag = 0;
    const unsigned te = sched(NUNIT - 1, SGRP(slab));
    for (int t = 0; t < NUNIT; ++t) {
      const char* s = frag_src(tab, t, lane);
      const unsigned ts = sched(t, SGRP(slab));
      if (!via_l2) {
        wait_until(t0 + ts);
        via_l2 = __builtin_amdgcn_readfirstlane(lds_read(ctl + 20));
        if (via_l2) {
          lag = (unsigned)rt() - (t0 + ts);
          lag = lag < 0xffffu ? lag : 0xffffu;
          if (lane == 0)
            __hip_atomic_store((GAS u64*)(myws + OFF_SW + slab * 128), ((u64)call << 32) | ((unsigned)t << 16) | lag, __ATOMIC_RELAXED, __HIP_MEMORY_SCOPE_AGENT);
        }
      }
      if (via_l2) {
        wait_until(t0 + lag + (ts + delta < te ? ts + delta : te));
        delta = delta + RAMP < DELTA ? delta + RAMP : DELTA;
        __builtin_amdgcn_s_setprio(LPRIO);
        __builtin_amdgcn_global_load_lds((const GAS unsigned*)s, (LAS unsigned*)(lds + t * 1024), 16, 0, LAUX);
        __builtin_amdgcn_s_setprio(0);
        wait_vm<LDEPTH>();
        if (t + 1 - LDEPTH > 0) lds_write(ctl + 4, t + 1 - LDEPTH);
      } else {
        __builtin_amdgcn_global_load_lds((const GAS unsigned*)s, (LAS unsigned*)(lds + t * 1024), 16, 0, 17);
        wait_vm<DEPTH>();
        if (t + 1 - DEPTH > 0) lds_write(ctl + 4, t + 1 - DEPTH);
      }
    }
    if (via_l2) drain<LDEPTH - 1>(ctl + 4); else drain<DEPTH - 1>(ctl + 4);
    wait_vm<0>();
    lds_write(ctl + 4, NUNIT);
    STAMP(2);
    return;
  }

  float scale0 = p.sx[0] * p.sw[0];
  unsigned out_lo = (unsigned)(size_t)p.out, out_hi = (unsigned)((size_t)p.out >> 32);
  asm volatile("" : "+v"(scale0), "+v"(out_lo), "+v"(out_hi));
  if (wave == 0) {
    const bool ok = wait_flags<XSLEEP>(myws + OFF_XFLAG + (bid & 7) * 256, 64, lane, 4, call, call, tmo);
    while (lds_read(ctl + 16) == 0) __builtin_amdgcn_s_sleep(1);
    STAMP(9);
    __builtin_amdgcn_fence(__ATOMIC_ACQUIRE, "");
    asm volatile("s_waitcnt vmcnt(0)" ::: "memory");
    if (lane == 0) __hip_atomic_fetch_add((GAS unsigned*)(myws + OFF_FC + (bid & 7) * 128), 1u, __ATOMIC_RELAXED, __HIP_MEMORY_SCOPE_AGENT);
    STAMP(3);
    const bool ok2 = wait_flags<2>(myws + OFF_FC + (bid & 7) * 128, 1, lane, 4, call * (NSLAB / 8), call, tmo);
    if (!ok || !ok2) lds_write(ctl + 12, 1);
    lds_write(ctl + 20, 1);
    STAMP(10);
    if (ld_agent(tmo) == call) lds_write(ctl + 12, 1);
  } else {
    while (lds_read(ctl + 20) == 0) __builtin_amdgcn_s_sleep(4);
  }
  asm volatile("" ::: "memory");

  const char* xa = myws + OFF_X + (size_t)(call & 1) * STAGE_BYTES + wave * 2048 + lane * 16;
  auto a_addr = [&](int t) { return xa + (size_t)((((j0 + t) & (NR - 1)) * NKS + (t >> 3)) << 14); };
  constexpr int PF = PFD;
  static_assert(NUNIT % PF == 0, "prefetch depth");
  i32x4 a[PF][2];
#pragma unroll
  for (int i = 0; i < PF; ++i) {
    const char* q = a_addr(i);
    a[i][0] = *(const GAS i32x4*)q;
    a[i][1] = *(const GAS i32x4*)(q + 1024);
  }
  const unsigned tc0 = (unsigned)rt();
  i32x16 acc0 = {}, acc1 = {};
  unsigned landed = 0;
  auto step = [&](int t, int i, bool prefetch) {
    while (landed <= (unsigned)t) {
      landed = __builtin_amdgcn_readfirstlane(lds_read(ctl + 4));
      if (landed <= (unsigned)t) __builtin_amdgcn_s_sleep(1);
    }
    const i32x4 b = *(const LAS i32x4*)(lds + t * 1024 + lane * 16);
    acc0 = __builtin_amdgcn_mfma_i32_32x32x32_i8(a[i][0], b, acc0, 0, 0, 0);
    acc1 = __builtin_amdgcn_mfma_i32_32x32x32_i8(a[i][1], b, acc1, 0, 0, 0);
    if (prefetch) {
      if (CPACE16) wait_until(tc0 + ((unsigned)(t * CPACE16) >> 4));
      const char* q = a_addr(t + PF);
      a[i][0] = *(const GAS i32x4*)q;
      a[i][1] = *(const GAS i32x4*)(q + 1024);
    }
  };
  for (int t0 = 0; t0 < NUNIT - PF; t0 += PF) {
#pragma unroll
    for (int i = 0; i < PF; ++i) step(t0 + i, i, true);
  }
#pragma unroll
  for (int i = 0; i < PF; ++i) step(NUNIT - PF + i, i, false);
  if (wave == 0) STAMP(4);
  STAMP(16 + wave);

  const float scale = lds_read(ctl + 12) != 0 ? __builtin_nanf("") : scale0;
  const int c31 = lane & 31;
  float* outp = (float*)(((size_t)(unsigned)__builtin_amdgcn_readfirstlane(out_hi) << 32) | (unsigned)__builtin_amdgcn_readfirstlane(out_lo));
  const __amdgpu_buffer_rsrc_t ors = __builtin_amdgcn_make_buffer_rsrc(outp, 0, MS * NDIM * 4, 0x00020000);
  const unsigned voff = (unsigned)(((wave * 64 + (lane >> 5) * 4) * NDIM + slab * 32 + (c31 & 7) * 4 + (c31 >> 3)) * 4);
  constexpr float LOG2E = 1.44269504088896340736f;
  const f32x2 s2 = {scale, scale}, t2 = {-scale * LOG2E, -scale * LOG2E}, one2 = {1.0f, 1.0f};
#pragma unroll
  for (int h = 0; h < 2; ++h) {
#pragma unroll
    for (int r = 0; r < 16; r += 2) {
      const f32x2 f = {(float)(h == 0 ? acc0[r] : acc1[r]), (float)(h == 0 ? acc0[r + 1] : acc1[r + 1])};
      const f32x2 y = f * s2, t = f * t2;
      const f32x2 e = {__builtin_amdgcn_exp2f(__builtin_amdgcn_fmed3f(t.x, -60.0f * LOG2E, 60.0f * LOG2E)),
                       __builtin_amdgcn_exp2f(__builtin_amdgcn_fmed3f(t.y, -60.0f * LOG2E, 60.0f * LOG2E))};
      const f32x2 d = e + one2;
      const f32x2 v = y * (f32x2){__builtin_amdgcn_rcpf(d.x), __builtin_amdgcn_rcpf(d.y)};
      const float v0 = v[0], v1 = v[1];
      __builtin_amdgcn_raw_buffer_store_b32(__float_as_uint(v0), ors, voff, (h * 32 + (r & 3) + 8 * (r >> 2)) * NDIM * 4, OUT_AUX);
      __builtin_amdgcn_raw_buffer_store_b32(__float_as_uint(v1), ors, voff, (h * 32 + ((r + 1) & 3) + 8 * (r >> 2)) * NDIM * 4, OUT_AUX);
    }
  }
}

}

extern "C" void kernel_launch(int rank, int nranks, ncclComm_t comm, hipStream_t stream,
                              void* const* d_in, const int* in_sizes, int n_in,
                              void* d_out, int out_size, const int* in_dtypes, int out_dtype,
                              void* const* d_out_all, void* const* d_ws_all, size_t ws_bytes) {
  (void)comm; (void)in_sizes; (void)in_dtypes; (void)out_dtype; (void)d_out_all;
  if (nranks != NR || n_in != 4 || out_size != MS * NDIM || ws_bytes < WS_NEED || rank < 0 || rank >= NR) {
    fprintf(stderr, "kernel_launch: unexpected configuration\n");
    return;
  }
  static bool attr_set[NR] = {};
  if (!attr_set[rank]) {
    HIP_CHECK(hipFuncSetAttribute((const void*)gemm_ag_kernel, hipFuncAttributeMaxDynamicSharedMemorySize, LDS_BYTES));
    attr_set[rank] = true;
  }
  Params p;
  p.x = (const char*)d_in[0];
  p.w = (const char*)d_in[1];
  p.sx = (const float*)d_in[2];
  p.sw = (const float*)d_in[3];
  p.out = (float*)d_out;
  for (int i = 0; i < NR; ++i) p.ws[i] = (char*)d_ws_all[i];
  p.myws = p.ws[rank];
  p.rank = rank;
  p.pad = 0;
  hipLaunchKernelGGL(gemm_ag_kernel, dim3(GRID), dim3(THREADS), LDS_BYTES, stream, p);
  HIP_CHECK(hipGetLastError());
}
```

```cpp
#include <hip/hip_runtime.h>
#include <rccl/rccl.h>
#include <cstdio>

#define HIP_CHECK(expr) do { hipError_t _e = (expr); if (_e != hipSuccess) { \
    fprintf(stderr, "HIP error %s at %s:%d\n", hipGetErrorString(_e), __FILE__, __LINE__); \
    return; } } while (0)

#define GAS __attribute__((address_space(1)))
#define LAS __attribute__((address_space(3)))
typedef unsigned u32x4 __attribute__((ext_vector_type(4)));
typedef int i32x4 __attribute__((ext_vector_type(4)));
typedef int i32x16 __attribute__((ext_vector_type(16)));
typedef float f32x2 __attribute__((ext_vector_type(2)));
typedef unsigned long long u64;

#define DEPTH 3
#define PACE16 440
#define STAG 0
#define XPACE 112
#define PFD 4
#define XDELAY 90
#define GODELAY 430
#define STAGGER 1
#define FASTPOLL 0
#define XWARM 0
#define XWDEPTH 8
#define FALLBACK 200
#define LDEPTH 2
#define PSKIP 3
#define PAUX 0
#define LAUX 0
#define CPACE16 0
#define RAMP 10
#define DELTA 60
#define PDEPTH 4
#define OUT_AUX 17
#define XSLEEP 8
#define LPRIO 3

namespace {

constexpr int NR = 8;
constexpr int NDIM = 4096;
constexpr int KS = 512;
constexpr int MS = 512;
constexpr int NSLAB = NDIM / 32;
constexpr int NKS = KS / 32;
constexpr int NUNIT = NR * NKS;
constexpr int N_WST = 64, N_STAGE = 128;
constexpr int GRID = N_STAGE + NSLAB;
constexpr int THREADS = 640;

constexpr size_t OFF_CNT = 0;
constexpr size_t OFF_TMO = OFF_CNT + GRID * 128;
constexpr size_t OFF_WCNT = OFF_TMO + 128;
constexpr size_t OFF_FC = OFF_WCNT + 8 * 128;
constexpr size_t OFF_GO = OFF_FC + 8 * 128;
constexpr size_t OFF_XFLAG = OFF_GO + 8 * 128;
constexpr size_t OFF_SW = 48 << 10;
constexpr size_t OFF_STAMP = 64 << 10;
constexpr size_t STAGE_BYTES = 2u << 20;
constexpr size_t OFF_X = 1u << 20;
constexpr size_t OFF_W = OFF_X + 2 * STAGE_BYTES;
constexpr size_t WS_NEED = OFF_W + 2 * STAGE_BYTES;
static_assert(OFF_XFLAG + 8 * 256 <= OFF_SW && OFF_SW + NSLAB * 128 <= OFF_STAMP, "layout");

constexpr int LDS_B = NUNIT * 1024;
constexpr int LDS_CTL = LDS_B;
constexpr int CTL_LANDED = 0;
constexpr int CTL_BAD = 8;
constexpr int CTL_GO = 12;
constexpr int CTL_T0 = 16;
constexpr int CTL_L2 = 20;
constexpr int CTL_LAG = 24;
constexpr int LDS_BYTES = LDS_B + 64;

constexpr u64 DEADLINE_TICKS = 50000000ull;

struct Params {
  const char* x;
  const char* w;
  const float* sx;
  const float* sw;
  float* out;
  char* myws;
  char* ws[NR];
  int rank;
  int pad;
};

__device__ __forceinline__ u64 rt() { return __builtin_amdgcn_s_memrealtime(); }
template <int N> __device__ __forceinline__ void wait_vm() { asm volatile("s_waitcnt vmcnt(%0)" :: "i"(N) : "memory"); }
__device__ __forceinline__ void store_wt(void* p, u32x4 v) {
  asm volatile("global_store_dwordx4 %0, %1, off sc0 sc1\n\ts_nop 1" :: "v"(p), "v"(v) : "memory");
}
__device__ __forceinline__ void lds_write(unsigned addr, unsigned v) {
  asm volatile("ds_write_b32 %0, %1" :: "v"(addr), "v"(v) : "memory");
}
template <int D> __device__ __forceinline__ void drain(unsigned addr) {
  if constexpr (D > 0) {
    wait_vm<D>();
    lds_write(addr, NUNIT / 2 - D);
    drain<D - 1>(addr);
  }
}
__device__ __forceinline__ unsigned lds_read(unsigned addr) {
  unsigned v;
  asm volatile("ds_read_b32 %0, %1\n\ts_waitcnt lgkmcnt(0)" : "=v"(v) : "v"(addr) : "memory");
  return v;
}
__device__ __forceinline__ unsigned ld_sys(const void* p) {
  return __hip_atomic_load((const GAS unsigned*)p, __ATOMIC_RELAXED, __HIP_MEMORY_SCOPE_SYSTEM);
}
__device__ __forceinline__ unsigned ld_agent(const void* p) {
  return __hip_atomic_load((const GAS unsigned*)p, __ATOMIC_RELAXED, __HIP_MEMORY_SCOPE_AGENT);
}
__device__ __forceinline__ void st_sys(void* p, unsigned v) {
  __hip_atomic_store((GAS unsigned*)p, v, __ATOMIC_RELAXED, __HIP_MEMORY_SCOPE_SYSTEM);
}
__device__ __forceinline__ void st_agent(void* p, unsigned v) {
  __hip_atomic_store((GAS unsigned*)p, v, __ATOMIC_RELAXED, __HIP_MEMORY_SCOPE_AGENT);
}
__device__ __forceinline__ unsigned poll_hint(const void* p) {
  unsigned v;
  asm volatile("s_load_dword %0, %1, 0x0 glc\n\ts_waitcnt lgkmcnt(0)" : "=s"(v) : "s"(p) : "memory");
  return v;
}
__device__ __forceinline__ unsigned perm(unsigned hi, unsigned lo, unsigned sel) { return __builtin_amdgcn_perm(hi, lo, sel); }
#define SGRP(slab) (((slab) >> 3) & 1)
#define PHASE(slab) (STAGGER ? (unsigned)((((slab) * 37) & 127) * PACE16) >> 11 : 0u)
__device__ __forceinline__ unsigned sched(int t, int grp) {
  constexpr unsigned T = NUNIT * PACE16;
  constexpr unsigned S = STAG * 16;
  if (STAG == 0) return (unsigned)(t * PACE16) >> 4;
  constexpr unsigned P0 = (T - S) / NUNIT;
  constexpr double r1 = 2.0 / PACE16 - 1.0 / P0;
  constexpr unsigned P1 = (unsigned)(1.0 / r1 + 0.5);
  constexpr int N1g = (int)((T - S) / P1);
  if (grp == 0) return (unsigned)(t * P0) >> 4;
  return t < N1g ? (unsigned)(t * P1) >> 4 : (unsigned)(N1g * P1 + (t - N1g) * (PACE16 / 2)) >> 4;
}
__device__ __forceinline__ void wait_until(unsigned t) {
  while ((int)((unsigned)rt() - t) < 0) __builtin_amdgcn_s_sleep(1);
}

#define STAMP(i) do { } while (0)

template <int SLEEP>
__device__ __forceinline__ bool wait_flags(const char* base, int n, int lane, int stride, unsigned target, unsigned call, char* tmo) {
  const u64 t0 = rt();
  const char* q = base + (lane < n ? lane : 0) * stride;
  for (unsigned it = 1;; ++it) {
    const unsigned v = ld_sys(q);
    if (__all(v >= target)) return true;
    if ((it & 63) == 0 && (ld_agent(tmo) == call || rt() - t0 > DEADLINE_TICKS)) { st_agent(tmo, call); return false; }
    __builtin_amdgcn_s_sleep(SLEEP);
  }
}

__device__ __forceinline__ bool wait_flags_fast(const char* base, int n, int lane, unsigned target, unsigned call, char* tmo) {
  const u64 t0 = rt();
  const char* q = base + (lane < n ? lane : 0) * 4;
  unsigned v0 = ld_sys(q);
  __builtin_amdgcn_s_sleep(4);
  unsigned v1 = ld_sys(q);
  __builtin_amdgcn_s_sleep(4);
  for (unsigned it = 1;; ++it) {
    const unsigned v2 = ld_sys(q);
    if (__all(v0 >= target)) return true;
    __builtin_amdgcn_s_sleep(4);
    v0 = ld_sys(q);
    if (__all(v1 >= target)) return true;
    __builtin_amdgcn_s_sleep(4);
    v1 = ld_sys(q);
    if (__all(v2 >= target)) return true;
    if ((it & 127) == 0 && (ld_agent(tmo) == call || rt() - t0 > DEADLINE_TICKS)) { st_agent(tmo, call); return false; }
    __builtin_amdgcn_s_sleep(4);
  }
}

__device__ __forceinline__ const char* frag_src(u64 tab, int t, int lane) {
  const unsigned lo = __builtin_amdgcn_readlane((unsigned)tab, t & 7), hi = __builtin_amdgcn_readlane((unsigned)(tab >> 32), t & 7);
  return (const char*)(((u64)hi << 32) | lo) + (t >> 3) * 1024 + lane * 16;
}

#define GO_WAIT wait_flags<1>(myws + OFF_GO + (bid & 7) * 128, 8, lane, 4, call, call, tmo)
__global__ void __launch_bounds__(THREADS) gemm_ag_kernel(Params p) {
  extern __shared__ __attribute__((aligned(16))) char lds[];
  const int tid = threadIdx.x;
  const int wave = __builtin_amdgcn_readfirstlane(tid >> 6);
  const int lane = tid & 63;
  const int bid = blockIdx.x;
  char* const myws = p.myws;
  char* const tmo = myws + OFF_TMO;
  const unsigned ctl = (unsigned)(size_t)(LAS char*)lds + LDS_CTL;

  char* const cntp = myws + OFF_CNT + bid * 128;
  unsigned cnt_v = ld_agent(cntp);
  if (wave == 0) STAMP(0);
#define GET_CALL() ({ asm volatile("" : "+v"(cnt_v) :: "memory"); (unsigned)__builtin_amdgcn_readfirstlane(cnt_v) + 1; })
  const unsigned t_start = (unsigned)rt();

  if (bid < N_WST) {
    const int G = bid >> 1, kh = bid & 1;
    const int pslab = bid + (wave & 1) * 64, pj0 = (pslab + (pslab >> 3)) & (NR - 1);
    u64 tab = (u64)p.ws[wave == 2 ? (lane & (NR - 1)) : ((pj0 + lane) & (NR - 1))];
    unsigned call;
    if (wave < 8) {
      const int ks = kh * 8 + wave;
      unsigned r[16];
      u32x4 v[4];
      const char* src = p.w + (size_t)(ks * 32 + (lane >> 5) * 16) * NDIM + G * 128 + (lane & 31) * 4;
#pragma unroll
      for (int i = 0; i < 16; ++i) r[i] = *(const GAS unsigned*)(src + (size_t)i * NDIM);
      call = GET_CALL();
#pragma unroll
      for (int g = 0; g < 4; ++g) {
        const unsigned lo01 = perm(r[4 * g + 1], r[4 * g], 0x05010400u), hi01 = perm(r[4 * g + 1], r[4 * g], 0x07030602u);
        const unsigned lo23 = perm(r[4 * g + 3], r[4 * g + 2], 0x05010400u), hi23 = perm(r[4 * g + 3], r[4 * g + 2], 0x07030602u);
        v[0][g] = perm(lo23, lo01, 0x05040100u);
        v[1][g] = perm(lo23, lo01, 0x07060302u);
        v[2][g] = perm(hi23, hi01, 0x05040100u);
        v[3][g] = perm(hi23, hi01, 0x07060302u);
      }
      const int slab = G * 4 + ((lane >> 3) & 3);
      char* dst = myws + OFF_W + (size_t)(call & 1) * STAGE_BYTES + (size_t)(slab * NKS + ks) * 1024 + ((lane & 7) + (lane >> 5) * 32) * 16;
#pragma unroll
      for (int q = 0; q < 4; ++q) store_wt(dst + q * 128, v[q]);
      wait_vm<0>();
    } else {
      call = GET_CALL();
    }
    asm volatile("s_barrier" ::: "memory");
    if (tid == 0) {
      __hip_atomic_fetch_add((GAS unsigned*)(myws + OFF_WCNT + (bid & 7) * 128), 1u, __ATOMIC_RELAXED, __HIP_MEMORY_SCOPE_AGENT);
      st_agent(cntp, call);
    }
    if (wave == 0) STAMP(1);
    if (bid == 0 && wave == 2) {
      const bool ok = wait_flags<1>(myws + OFF_WCNT, 8, lane, 128, call * (N_WST / 8), call, tmo);
      wait_until(t_start + GODELAY);
      if (ok) st_sys((char*)tab + OFF_GO + (lane >> 3) * 128 + p.rank * 4, call);
      STAMP(2);
    }
    if (wave < 2) {
      tab += OFF_W + (size_t)(call & 1) * STAGE_BYTES + (size_t)pslab * NKS * 1024;
      if (!GO_WAIT) return;
      const unsigned t0 = (unsigned)rt() + PHASE(pslab);
      if (wave == 0) STAMP(3);
      u64 m;
      {
        const u64 tw0 = rt();
        for (unsigned it = 1;; ++it) {
          m = __hip_atomic_load((const GAS u64*)(myws + OFF_SW + pslab * 128), __ATOMIC_RELAXED, __HIP_MEMORY_SCOPE_AGENT);
          if (__builtin_amdgcn_readfirstlane((unsigned)(m >> 32)) == call) break;
          if ((it & 63) == 0 && (ld_agent(tmo) == call || rt() - tw0 > DEADLINE_TICKS)) { st_agent(tmo, call); return; }
          __builtin_amdgcn_s_sleep(2);
        }
      }
      const unsigned mw = __builtin_amdgcn_readfirstlane((unsigned)m);
      const unsigned tp = t0 + (mw & 0xffffu);
      if (wave == 0) STAMP(5);
      for (int t = (int)(mw >> 16) + PSKIP; t < NUNIT; ++t) {
        if (((pj0 + t) & (NR - 1)) == p.rank) continue;
        const char* s = frag_src(tab, t, lane);
        wait_until(tp + sched(t, SGRP(pslab)));
        __builtin_amdgcn_global_load_lds((const GAS unsigned*)s, (LAS unsigned*)(lds + ((wave * 16 + (t & 15)) << 10)), 16, 0, PAUX);
        if (lane == 0) st_agent(myws + OFF_SW + pslab * 128 + 64, (call << 8) | (unsigned)(t + 1));
        wait_vm<2 * PDEPTH>();
      }
      wait_vm<0>();
      if (wave == 0) STAMP(4);
    }
    return;
  }
  if (bid < N_STAGE) {
    const int idx = bid - N_WST;
    const int dst_rank = (p.rank + (idx >> 3)) & (NR - 1), part = idx & 7;
    char* const dws = p.ws[dst_rank];
    unsigned call = 0;
    if (wave < 8) {
      const int rtile = part * 2 + (wave & 1), ks0 = (wave >> 1) * 4;
      u32x4 v[4];
      const char* src = p.x + (size_t)(dst_rank * MS + rtile * 32 + (lane & 31)) * KS + ks0 * 32 + (lane >> 5) * 16;
#pragma unroll
      for (int i = 0; i < 4; ++i) v[i] = *(const GAS u32x4*)(src + i * 32);
      call = GET_CALL();
      char* xdst = dws + OFF_X + (size_t)(call & 1) * STAGE_BYTES + (size_t)((p.rank * NKS + ks0) * 16 + rtile) * 1024 + lane * 16;
#pragma unroll
      for (int i = 0; i < 4; ++i) {
        wait_until(t_start + XDELAY + ((wave * XPACE) >> 3) + i * XPACE);
        store_wt(xdst + i * 16384, v[i]);
      }
    }
    wait_vm<0>();
    asm volatile("s_barrier" ::: "memory");
    if (tid < 8) st_sys(dws + OFF_XFLAG + tid * 256 + (p.rank * 8 + part) * 4, call);
    if (tid == 0) st_agent(cntp, call);
    if (wave == 0) STAMP(2);
    return;
  }

  const int slab = bid - N_STAGE;
  const int j0 = (slab + (slab >> 3)) & (NR - 1);
  u64 tab = 0;
  if (wave >= 8) tab = (u64)p.ws[(j0 + lane) & (NR - 1)];
  const unsigned call = GET_CALL();
  if (tid < 16) lds_write(ctl + tid * 4, 0);
  asm volatile("s_waitcnt lgkmcnt(0)\n\ts_barrier" ::: "memory");
  if (tid == 0) st_agent(cntp, call);

  if (wave >= 8) {
    const int par = wave - 8;
    const unsigned cnt_addr = ctl + CTL_LANDED + par * 4;
    tab += OFF_W + (size_t)(call & 1) * STAGE_BYTES + (size_t)slab * NKS * 1024;
    unsigned t0;
    if (par == 0) {
      const bool ok = GO_WAIT;
      t0 = (unsigned)rt() + PHASE(slab);
      STAMP(1);
      if (!ok) {
        lds_write(ctl + CTL_BAD, 1);
        lds_write(ctl + CTL_T0, 0);
        lds_write(ctl + CTL_LANDED, NUNIT / 2);
        lds_write(ctl + CTL_LANDED + 4, NUNIT / 2);
        lds_write(ctl + CTL_GO, 2);
        return;
      }
      lds_write(ctl + CTL_T0, t0);
      lds_write(ctl + CTL_GO, 1);
    } else {
      unsigned g;
      while ((g = __builtin_amdgcn_readfirstlane(lds_read(ctl + CTL_GO))) == 0) __builtin_amdgcn_s_sleep(1);
      if (g == 2) return;
      t0 = __builtin_amdgcn_readfirstlane(lds_read(ctl + CTL_T0));
    }
    int t = par;
    unsigned lag1 = 0;
    for (; t < NUNIT; t += 2) {
      const char* s = frag_src(tab, t, lane);
      wait_until(t0 + sched(t, SGRP(slab)));
      lag1 = __builtin_amdgcn_readfirstlane(lds_read(ctl + (par == 0 ? CTL_L2 : CTL_LAG)));
      if (lag1) break;
      __builtin_amdgcn_global_load_lds((const GAS unsigned*)s, (LAS unsigned*)(lds + t * 1024), 16, 0, 17);
      wait_vm<DEPTH>();
      lds_write(cnt_addr, max((t >> 1) + 1 - DEPTH, 0));
    }
    if (t < NUNIT) {
      if (par == 0) {
        unsigned lag = (unsigned)rt() - (t0 + sched(t, SGRP(slab)));
        lag = lag < 0xfffeu ? lag : 0xfffeu;
        if (lane == 0)
          __hip_atomic_store((GAS u64*)(myws + OFF_SW + slab * 128), ((u64)call << 32) | ((unsigned)t << 16) | lag, __ATOMIC_RELAXED, __HIP_MEMORY_SCOPE_AGENT);
        lag1 = lag + 1;
        lds_write(ctl + CTL_LAG, lag1);
      }
      const unsigned tl = t0 + lag1 - 1, te = sched(NUNIT - 1, SGRP(slab));
      const char* prog = myws + OFF_SW + slab * 128 + 64;
      const int t_first = t + PSKIP + 1;
      unsigned v = 0, delta = 0;
      for (; t < NUNIT; t += 2) {
        const char* s = frag_src(tab, t, lane);
        const unsigned ts = sched(t, SGRP(slab)) + delta;
        delta = delta + 2 * RAMP < DELTA ? delta + 2 * RAMP : DELTA;
        const unsigned target = (call << 8) | (unsigned)(t + 1);
        const bool follow = t >= t_first && ((j0 + t) & (NR - 1)) != p.rank;
        if (follow && v < target) v = poll_hint(prog);
        wait_until(tl + (ts < te ? ts : te));
        if (follow) {
          unsigned tchg = (unsigned)rt();
          while (v < target) {
            const unsigned w = poll_hint(prog), now = (unsigned)rt();
            if (w != v) { v = w; tchg = now; }
            else if (now - tchg > FALLBACK) break;
          }
        }
        __builtin_amdgcn_s_setprio(LPRIO);
        __builtin_amdgcn_global_load_lds((const GAS unsigned*)s, (LAS unsigned*)(lds + t * 1024), 16, 0, LAUX);
        __builtin_amdgcn_s_setprio(0);
        wait_vm<LDEPTH>();
        lds_write(cnt_addr, max((t >> 1) + 1 - LDEPTH, 0));
      }
      drain<LDEPTH - 1>(cnt_addr);
    }
    wait_vm<0>();
    lds_write(cnt_addr, NUNIT / 2);
    if (par == 1) STAMP(2);
    return;
  }

  float scale0 = p.sx[0] * p.sw[0];
  unsigned out_lo = (unsigned)(size_t)p.out, out_hi = (unsigned)((size_t)p.out >> 32);
  asm volatile("" : "+v"(scale0), "+v"(out_lo), "+v"(out_hi));
  if (wave == 0) {
    const bool ok = wait_flags<XSLEEP>(myws + OFF_XFLAG + (bid & 7) * 256, 64, lane, 4, call, call, tmo);
    while (lds_read(ctl + CTL_GO) == 0) __builtin_amdgcn_s_sleep(1);
    STAMP(9);
    __builtin_amdgcn_fence(__ATOMIC_ACQUIRE, "");
    asm volatile("s_waitcnt vmcnt(0)" ::: "memory");
    if (lane == 0) __hip_atomic_fetch_add((GAS unsigned*)(myws + OFF_FC + (bid & 7) * 128), 1u, __ATOMIC_RELAXED, __HIP_MEMORY_SCOPE_AGENT);
    STAMP(3);
    const bool ok2 = wait_flags<2>(myws + OFF_FC + (bid & 7) * 128, 1, lane, 4, call * (NSLAB / 8), call, tmo);
    if (!ok || !ok2) lds_write(ctl + CTL_BAD, 1);
    lds_write(ctl + CTL_L2, 1);
    STAMP(10);
    if (ld_agent(tmo) == call) lds_write(ctl + CTL_BAD, 1);
  } else {
    while (lds_read(ctl + CTL_L2) == 0) __builtin_amdgcn_s_sleep(4);
  }
  asm volatile("" ::: "memory");

  const char* xa = myws + OFF_X + (size_t)(call & 1) * STAGE_BYTES + wave * 2048 + lane * 16;
  auto a_addr = [&](int t) { return xa + (size_t)((((j0 + t) & (NR - 1)) * NKS + (t >> 3)) << 14); };
  constexpr int PF = PFD;
  static_assert(NUNIT % PF == 0 && PF % 2 == 0, "prefetch depth");
  i32x4 a[PF][2];
#pragma unroll
  for (int i = 0; i < PF; ++i) {
    const char* q = a_addr(i);
    a[i][0] = *(const GAS i32x4*)q;
    a[i][1] = *(const GAS i32x4*)(q + 1024);
  }
  const unsigned tc0 = (unsigned)rt();
  i32x16 acc0 = {}, acc1 = {};
  unsigned landed[2] = {0, 0};
  auto step = [&](int t, int i, bool prefetch) {
    while (landed[i & 1] <= (unsigned)(t >> 1)) {
      landed[i & 1] = __builtin_amdgcn_readfirstlane(lds_read(ctl + CTL_LANDED + (i & 1) * 4));
      if (landed[i & 1] <= (unsigned)(t >> 1)) __builtin_amdgcn_s_sleep(1);
    }
    const i32x4 b = *(const LAS i32x4*)(lds + t * 1024 + lane * 16);
    acc0 = __builtin_amdgcn_mfma_i32_32x32x32_i8(a[i][0], b, acc0, 0, 0, 0);
    acc1 = __builtin_amdgcn_mfma_i32_32x32x32_i8(a[i][1], b, acc1, 0, 0, 0);
    if (prefetch) {
      if (CPACE16) wait_until(tc0 + ((unsigned)(t * CPACE16) >> 4));
      const char* q = a_addr(t + PF);
      a[i][0] = *(const GAS i32x4*)q;
      a[i][1] = *(const GAS i32x4*)(q + 1024);
    }
  };
  for (int t0 = 0; t0 < NUNIT - PF; t0 += PF) {
#pragma unroll
    for (int i = 0; i < PF; ++i) step(t0 + i, i, true);
  }
#pragma unroll
  for (int i = 0; i < PF; ++i) step(NUNIT - PF + i, i, false);
  if (wave == 0) STAMP(4);
  STAMP(16 + wave);

  const float scale = lds_read(ctl + CTL_BAD) != 0 ? __builtin_nanf("") : scale0;
  const int c31 = lane & 31;
  float* outp = (float*)(((size_t)(unsigned)__builtin_amdgcn_readfirstlane(out_hi) << 32) | (unsigned)__builtin_amdgcn_readfirstlane(out_lo));
  const __amdgpu_buffer_rsrc_t ors = __builtin_amdgcn_make_buffer_rsrc(outp, 0, MS * NDIM * 4, 0x00020000);
  const unsigned voff = (unsigned)(((wave * 64 + (lane >> 5) * 4) * NDIM + slab * 32 + (c31 & 7) * 4 + (c31 >> 3)) * 4);
  constexpr float LOG2E = 1.44269504088896340736f;
  const f32x2 s2 = {scale, scale}, t2 = {-scale * LOG2E, -scale * LOG2E}, one2 = {1.0f, 1.0f};
#pragma unroll
  for (int h = 0; h < 2; ++h) {
#pragma unroll
    for (int r = 0; r < 16; r += 2) {
      const f32x2 f = {(float)(h == 0 ? acc0[r] : acc1[r]), (float)(h == 0 ? acc0[r + 1] : acc1[r + 1])};
      const f32x2 y = f * s2, t = f * t2;
      const f32x2 e = {__builtin_amdgcn_exp2f(__builtin_amdgcn_fmed3f(t.x, -60.0f * LOG2E, 60.0f * LOG2E)),
                       __builtin_amdgcn_exp2f(__builtin_amdgcn_fmed3f(t.y, -60.0f * LOG2E, 60.0f * LOG2E))};
      const f32x2 d = e + one2;
      const f32x2 v = y * (f32x2){__builtin_amdgcn_rcpf(d.x), __builtin_amdgcn_rcpf(d.y)};
      const float v0 = v[0], v1 = v[1];
      __builtin_amdgcn_raw_buffer_store_b32(__float_as_uint(v0), ors, voff, (h * 32 + (r & 3) + 8 * (r >> 2)) * NDIM * 4, OUT_AUX);
      __builtin_amdgcn_raw_buffer_store_b32(__float_as_uint(v1), ors, voff, (h * 32 + ((r + 1) & 3) + 8 * (r >> 2)) * NDIM * 4, OUT_AUX);
    }
  }
}

}

extern "C" void kernel_launch(int rank, int nranks, ncclComm_t comm, hipStream_t stream,
                              void* const* d_in, const int* in_sizes, int n_in,
                              void* d_out, int out_size, const int* in_dtypes, int out_dtype,
                              void* const* d_out_all, void* const* d_ws_all, size_t ws_bytes) {
  (void)comm; (void)in_sizes; (void)in_dtypes; (void)out_dtype; (void)d_out_all;
  if (nranks != NR || n_in != 4 || out_size != MS * NDIM || ws_bytes < WS_NEED || rank < 0 || rank >= NR) {
    fprintf(stderr, "kernel_launch: unexpected configuration\n");
    return;
  }
  static bool attr_set[NR] = {};
  if (!attr_set[rank]) {
    HIP_CHECK(hipFuncSetAttribute((const void*)gemm_ag_kernel, hipFuncAttributeMaxDynamicSharedMemorySize, LDS_BYTES));
    attr_set[rank] = true;
  }
  Params p;
  p.x = (const char*)d_in[0];
  p.w = (const char*)d_in[1];
  p.sx = (const float*)d_in[2];
  p.sw = (const float*)d_in[3];
  p.out = (float*)d_out;
  for (int i = 0; i < NR; ++i) p.ws[i] = (char*)d_ws_all[i];
  p.myws = p.ws[rank];
  p.rank = rank;
  p.pad = 0;
  hipLaunchKernelGGL(gemm_ag_kernel, dim3(GRID), dim3(THREADS), LDS_BYTES, stream, p);
  HIP_CHECK(hipGetLastError());
}
```

```cpp
#include <hip/hip_runtime.h>
#include <rccl/rccl.h>
#include <cstdio>

#define HIP_CHECK(expr) do { hipError_t _e = (expr); if (_e != hipSuccess) { \
    fprintf(stderr, "HIP error %s at %s:%d\n", hipGetErrorString(_e), __FILE__, __LINE__); \
    return; } } while (0)

#define GAS __attribute__((address_space(1)))
#define LAS __attribute__((address_space(3)))
typedef unsigned u32x4 __attribute__((ext_vector_type(4)));
typedef int i32x4 __attribute__((ext_vector_type(4)));
typedef int i32x16 __attribute__((ext_vector_type(16)));
typedef float f32x2 __attribute__((ext_vector_type(2)));
typedef unsigned long long u64;

#define PACE16 440
#define XDELAY 90
#define XPACE 112
#define GODELAY 430
#define DELTA 60
constexpr int DEPTH = 3;
constexpr int LDEPTH = 2;
constexpr int PDEPTH = 4;
constexpr int PSKIP = 3;
constexpr int RAMP = 10;
constexpr int FALLBACK = 200;
constexpr int PF = 4;

namespace {

constexpr int NR = 8;
constexpr int NDIM = 4096;
constexpr int KS = 512;
constexpr int MS = 512;
constexpr int NSLAB = NDIM / 32;
constexpr int NKS = KS / 32;
constexpr int NUNIT = NR * NKS;
constexpr int N_WST = 64, N_STAGE = 128;
constexpr int GRID = N_STAGE + NSLAB;
constexpr int THREADS = 640;

constexpr size_t OFF_CNT = 0;
constexpr size_t OFF_TMO = OFF_CNT + GRID * 128;
constexpr size_t OFF_WCNT = OFF_TMO + 128;
constexpr size_t OFF_FC = OFF_WCNT + 8 * 128;
constexpr size_t OFF_GO = OFF_FC + 8 * 128;
constexpr size_t OFF_XFLAG = OFF_GO + 8 * 128;
constexpr size_t OFF_SW = 48 << 10;
constexpr size_t OFF_STAMP = 64 << 10;
constexpr size_t STAGE_BYTES = 2u << 20;
constexpr size_t OFF_X = 1u << 20;
constexpr size_t OFF_W = OFF_X + 2 * STAGE_BYTES;
constexpr size_t WS_NEED = OFF_W + 2 * STAGE_BYTES;
static_assert(OFF_XFLAG + 8 * 256 <= OFF_SW && OFF_SW + NSLAB * 128 <= OFF_STAMP, "layout");

constexpr int LDS_B = NUNIT * 1024;
constexpr int LDS_CTL = LDS_B;
constexpr int CTL_LANDED = 0;
constexpr int CTL_BAD = 8;
constexpr int CTL_GO = 12;
constexpr int CTL_T0 = 16;
constexpr int CTL_L2 = 20;
constexpr int CTL_LAG = 24;
constexpr int LDS_BYTES = LDS_B + 64;

constexpr u64 DEADLINE_TICKS = 50000000ull;

struct Params {
  const char* x;
  const char* w;
  const float* sx;
  const float* sw;
  float* out;
  char* myws;
  char* ws[NR];
  int rank;
  int pad;
};

__device__ __forceinline__ u64 rt() { return __builtin_amdgcn_s_memrealtime(); }
template <int N> __device__ __forceinline__ void wait_vm() { asm volatile("s_waitcnt vmcnt(%0)" :: "i"(N) : "memory"); }
__device__ __forceinline__ void store_wt(void* p, u32x4 v) {
  asm volatile("global_store_dwordx4 %0, %1, off sc0 sc1\n\ts_nop 1" :: "v"(p), "v"(v) : "memory");
}
__device__ __forceinline__ void lds_write(unsigned addr, unsigned v) {
  asm volatile("ds_write_b32 %0, %1" :: "v"(addr), "v"(v) : "memory");
}
template <int D> __device__ __forceinline__ void drain(unsigned addr) {
  if constexpr (D > 0) {
    wait_vm<D>();
    lds_write(addr, NUNIT / 2 - D);
    drain<D - 1>(addr);
  }
}
__device__ __forceinline__ unsigned lds_read(unsigned addr) {
  unsigned v;
  asm volatile("ds_read_b32 %0, %1\n\ts_waitcnt lgkmcnt(0)" : "=v"(v) : "v"(addr) : "memory");
  return v;
}
__device__ __forceinline__ unsigned ld_sys(const void* p) {
  return __hip_atomic_load((const GAS unsigned*)p, __ATOMIC_RELAXED, __HIP_MEMORY_SCOPE_SYSTEM);
}
__device__ __forceinline__ unsigned ld_agent(const void* p) {
  return __hip_atomic_load((const GAS unsigned*)p, __ATOMIC_RELAXED, __HIP_MEMORY_SCOPE_AGENT);
}
__device__ __forceinline__ void st_sys(void* p, unsigned v) {
  __hip_atomic_store((GAS unsigned*)p, v, __ATOMIC_RELAXED, __HIP_MEMORY_SCOPE_SYSTEM);
}
__device__ __forceinline__ void st_agent(void* p, unsigned v) {
  __hip_atomic_store((GAS unsigned*)p, v, __ATOMIC_RELAXED, __HIP_MEMORY_SCOPE_AGENT);
}
__device__ __forceinline__ unsigned poll_hint(const void* p) {
  unsigned v;
  asm volatile("s_load_dword %0, %1, 0x0 glc\n\ts_waitcnt lgkmcnt(0)" : "=s"(v) : "s"(p) : "memory");
  return v;
}
__device__ __forceinline__ unsigned perm(unsigned hi, unsigned lo, unsigned sel) { return __builtin_amdgcn_perm(hi, lo, sel); }
__device__ __forceinline__ unsigned sched(int t) { return (unsigned)(t * PACE16) >> 4; }
#define PHOFF 27
__device__ __forceinline__ unsigned phase(int slab) { return ((unsigned)(((slab * 37) & 127) * PACE16) >> 11) - PHOFF; }
__device__ __forceinline__ void wait_until(unsigned t) {
  while ((int)((unsigned)rt() - t) < 0) __builtin_amdgcn_s_sleep(1);
}

#define STAMP(i) do { } while (0)

template <int SLEEP>
__device__ __forceinline__ bool wait_flags(const char* base, int n, int lane, int stride, unsigned target, unsigned call, char* tmo) {
  const u64 t0 = rt();
  const char* q = base + (lane < n ? lane : 0) * stride;
  for (unsigned it = 1;; ++it) {
    const unsigned v = ld_sys(q);
    if (__all(v >= target)) return true;
    if ((it & 63) == 0 && (ld_agent(tmo) == call || rt() - t0 > DEADLINE_TICKS)) { st_agent(tmo, call); return false; }
    __builtin_amdgcn_s_sleep(SLEEP);
  }
}

__device__ __forceinline__ const char* frag_src(u64 tab, int t, int lane) {
  const unsigned lo = __builtin_amdgcn_readlane((unsigned)tab, t & 7), hi = __builtin_amdgcn_readlane((unsigned)(tab >> 32), t & 7);
  return (const char*)(((u64)hi << 32) | lo) + (t >> 3) * 1024 + lane * 16;
}

__global__ void __launch_bounds__(THREADS) gemm_ag_kernel(Params p) {
  extern __shared__ __attribute__((aligned(16))) char lds[];
  const int tid = threadIdx.x;
  const int wave = __builtin_amdgcn_readfirstlane(tid >> 6);
  const int lane = tid & 63;
  const int bid = blockIdx.x;
  char* const myws = p.myws;
  char* const tmo = myws + OFF_TMO;
  const unsigned ctl = (unsigned)(size_t)(LAS char*)lds + LDS_CTL;

  char* const cntp = myws + OFF_CNT + bid * 128;
  unsigned cnt_v = ld_agent(cntp);
  if (wave == 0) STAMP(0);
#define GET_CALL() ({ asm volatile("" : "+v"(cnt_v) :: "memory"); (unsigned)__builtin_amdgcn_readfirstlane(cnt_v) + 1; })
  const unsigned t_start = (unsigned)rt();

  const int sb = bid ^ 64;
  if (bid < N_STAGE && sb < N_WST) {
    const int G = sb >> 1, kh = sb & 1;
    const int pslab = sb + (wave & 1) * 64, pj0 = (pslab + (pslab >> 3)) & (NR - 1);
    u64 tab = (u64)p.ws[wave == 2 ? (lane & (NR - 1)) : ((pj0 + lane) & (NR - 1))];
    unsigned call;
    if (wave < 8) {
      const int ks = kh * 8 + wave;
      unsigned r[16];
      u32x4 v[4];
      const char* src = p.w + (size_t)(ks * 32 + (lane >> 5) * 16) * NDIM + G * 128 + (lane & 31) * 4;
#pragma unroll
      for (int i = 0; i < 16; ++i) r[i] = *(const GAS unsigned*)(src + (size_t)i * NDIM);
      call = GET_CALL();
#pragma unroll
      for (int g = 0; g < 4; ++g) {
        const unsigned lo01 = perm(r[4 * g + 1], r[4 * g], 0x05010400u), hi01 = perm(r[4 * g + 1], r[4 * g], 0x07030602u);
        const unsigned lo23 = perm(r[4 * g + 3], r[4 * g + 2], 0x05010400u), hi23 = perm(r[4 * g + 3], r[4 * g + 2], 0x07030602u);
        v[0][g] = perm(lo23, lo01, 0x05040100u);
        v[1][g] = perm(lo23, lo01, 0x07060302u);
        v[2][g] = perm(hi23, hi01, 0x05040100u);
        v[3][g] = perm(hi23, hi01, 0x07060302u);
      }
      const int slab = G * 4 + ((lane >> 3) & 3);
      char* dst = myws + OFF_W + (size_t)(call & 1) * STAGE_BYTES + (size_t)(slab * NKS + ks) * 1024 + ((lane & 7) + (lane >> 5) * 32) * 16;
#pragma unroll
      for (int q = 0; q < 4; ++q) store_wt(dst + q * 128, v[q]);
      wait_vm<0>();
    } else {
      call = GET_CALL();
    }
    asm volatile("s_barrier" ::: "memory");
    if (tid == 0) {
      __hip_atomic_fetch_add((GAS unsigned*)(myws + OFF_WCNT + (bid & 7) * 128), 1u, __ATOMIC_RELAXED, __HIP_MEMORY_SCOPE_AGENT);
      st_agent(cntp, call);
    }
    if (wave == 0) STAMP(1);
    if (sb == 0 && wave == 2) {
      const bool ok = wait_flags<1>(myws + OFF_WCNT, 8, lane, 128, call * (N_WST / 8), call, tmo);
      wait_until(t_start + GODELAY);
      if (ok) st_sys((char*)tab + OFF_GO + (lane >> 3) * 128 + p.rank * 4, call);
      STAMP(2);
    }
    if (wave < 2) {
      tab += OFF_W + (size_t)(call & 1) * STAGE_BYTES + (size_t)pslab * NKS * 1024;
      if (!wait_flags<1>(myws + OFF_GO + (bid & 7) * 128, 8, lane, 4, call, call, tmo)) return;
      const unsigned t0 = (unsigned)rt() + phase(pslab);
      if (wave == 0) STAMP(3);
      u64 m;
      {
        const u64 tw0 = rt();
        for (unsigned it = 1;; ++it) {
          m = __hip_atomic_load((const GAS u64*)(myws + OFF_SW + pslab * 128), __ATOMIC_RELAXED, __HIP_MEMORY_SCOPE_AGENT);
          if (__builtin_amdgcn_readfirstlane((unsigned)(m >> 32)) == call) break;
          if ((it & 63) == 0 && (ld_agent(tmo) == call || rt() - tw0 > DEADLINE_TICKS)) return;
          __builtin_amdgcn_s_sleep(2);
        }
      }
      const unsigned mw = __builtin_amdgcn_readfirstlane((unsigned)m);
      const unsigned tp = t0 + (mw & 0xffffu);
      if (wave == 0) STAMP(5);
      for (int t = (int)(mw >> 16) + PSKIP; t < NUNIT; ++t) {
        if (((pj0 + t) & (NR - 1)) == p.rank) continue;
        const char* s = frag_src(tab, t, lane);
        wait_until(tp + sched(t));
        __builtin_amdgcn_global_load_lds((const GAS unsigned*)s, (LAS unsigned*)(lds + ((wave * 16 + (t & 15)) << 10)), 16, 0, 0);
        if (lane == 0) st_agent(myws + OFF_SW + pslab * 128 + 64, (call << 8) | (unsigned)(t + 1));
        wait_vm<2 * PDEPTH>();
      }
      wait_vm<0>();
      if (wave == 0) STAMP(4);
    }
    return;
  }
  if (bid < N_STAGE) {
    const int idx = sb - N_WST;
    const int dst_rank = (p.rank + (idx >> 3)) & (NR - 1), part = idx & 7;
    char* const dws = p.ws[dst_rank];
    unsigned call = 0;
    if (wave < 8) {
      const int rtile = part * 2 + (wave & 1), ks0 = (wave >> 1) * 4;
      u32x4 v[4];
      const char* src = p.x + (size_t)(dst_rank * MS + rtile * 32 + (lane & 31)) * KS + ks0 * 32 + (lane >> 5) * 16;
#pragma unroll
      for (int i = 0; i < 4; ++i) v[i] = *(const GAS u32x4*)(src + i * 32);
      call = GET_CALL();
      char* xdst = dws + OFF_X + (size_t)(call & 1) * STAGE_BYTES + (size_t)((p.rank * NKS + ks0) * 16 + rtile) * 1024 + lane * 16;
#pragma unroll
      for (int i = 0; i < 4; ++i) {
        wait_until(t_start + XDELAY + ((wave * XPACE) >> 3) + i * XPACE);
        store_wt(xdst + i * 16384, v[i]);
      }
    }
    wait_vm<0>();
    asm volatile("s_barrier" ::: "memory");
    if (tid < 8) st_sys(dws + OFF_XFLAG + tid * 256 + (p.rank * 8 + part) * 4, call);
    if (tid == 0) st_agent(cntp, call);
    if (wave == 0) STAMP(2);
    return;
  }

  const int slab = bid - N_STAGE;
  const int j0 = (slab + (slab >> 3)) & (NR - 1);
  u64 tab = 0;
  if (wave >= 8) tab = (u64)p.ws[(j0 + lane) & (NR - 1)];
  const unsigned call = GET_CALL();
  if (tid < 16) lds_write(ctl + tid * 4, 0);
  asm volatile("s_waitcnt lgkmcnt(0)\n\ts_barrier" ::: "memory");
  if (tid == 0) st_agent(cntp, call);

  if (wave >= 8) {
    const int par = wave - 8;
    const unsigned cnt_addr = ctl + CTL_LANDED + par * 4;
    tab += OFF_W + (size_t)(call & 1) * STAGE_BYTES + (size_t)slab * NKS * 1024;
    unsigned t0;
    if (par == 0) {
      const bool ok = wait_flags<1>(myws + OFF_GO + (bid & 7) * 128, 8, lane, 4, call, call, tmo);
      t0 = (unsigned)rt() + phase(slab);
      STAMP(1);
      if (!ok) {
        lds_write(ctl + CTL_BAD, 1);
        lds_write(ctl + CTL_T0, 0);
        lds_write(ctl + CTL_LANDED, NUNIT / 2);
        lds_write(ctl + CTL_LANDED + 4, NUNIT / 2);
        lds_write(ctl + CTL_GO, 2);
        if (lane == 0)
          __hip_atomic_store((GAS u64*)(myws + OFF_SW + slab * 128), ((u64)call << 32) | ((unsigned)NUNIT << 16), __ATOMIC_RELAXED, __HIP_MEMORY_SCOPE_AGENT);
        return;
      }
      lds_write(ctl + CTL_T0, t0);
      lds_write(ctl + CTL_GO, 1);
    } else {
      unsigned g;
      while ((g = __builtin_amdgcn_readfirstlane(lds_read(ctl + CTL_GO))) == 0) __builtin_amdgcn_s_sleep(1);
      if (g == 2) return;
      t0 = __builtin_amdgcn_readfirstlane(lds_read(ctl + CTL_T0));
    }
    int t = par;
    unsigned lag1 = 0;
    for (; t < NUNIT; t += 2) {
      const char* s = frag_src(tab, t, lane);
      wait_until(t0 + sched(t));
      lag1 = __builtin_amdgcn_readfirstlane(lds_read(ctl + (par == 0 ? CTL_L2 : CTL_LAG)));
      if (lag1) break;
      __builtin_amdgcn_global_load_lds((const GAS unsigned*)s, (LAS unsigned*)(lds + t * 1024), 16, 0, 17);
      wait_vm<DEPTH>();
      lds_write(cnt_addr, max((t >> 1) + 1 - DEPTH, 0));
    }
    if (par == 0) {
      unsigned lag = (unsigned)rt() - (t0 + sched(t));
      lag = lag < 0xfffeu ? lag : 0xfffeu;
      if (lane == 0)
        __hip_atomic_store((GAS u64*)(myws + OFF_SW + slab * 128), ((u64)call << 32) | ((unsigned)t << 16) | lag, __ATOMIC_RELAXED, __HIP_MEMORY_SCOPE_AGENT);
      lag1 = lag + 1;
      if (t < NUNIT) lds_write(ctl + CTL_LAG, lag1);
    }
    if (t < NUNIT) {
      const unsigned tl = t0 + lag1 - 1, te = sched(NUNIT - 1);
      const char* prog = myws + OFF_SW + slab * 128 + 64;
      const int t_first = t + PSKIP + 1;
      unsigned v = 0, delta = 0;
      for (; t < NUNIT; t += 2) {
        const char* s = frag_src(tab, t, lane);
        const unsigned ts = sched(t) + delta;
        delta = delta + 2 * RAMP < DELTA ? delta + 2 * RAMP : DELTA;
        const unsigned target = (call << 8) | (unsigned)(t + 1);
        const bool follow = t >= t_first && ((j0 + t) & (NR - 1)) != p.rank;
        if (follow && v < target) v = poll_hint(prog);
        wait_until(tl + (ts < te ? ts : te));
        if (follow) {
          unsigned tchg = (unsigned)rt();
          while (v < target) {
            const unsigned w = poll_hint(prog), now = (unsigned)rt();
            if (w != v) { v = w; tchg = now; }
            else if (now - tchg > FALLBACK) break;
          }
        }
        __builtin_amdgcn_s_setprio(3);
        __builtin_amdgcn_global_load_lds((const GAS unsigned*)s, (LAS unsigned*)(lds + t * 1024), 16, 0, 0);
        __builtin_amdgcn_s_setprio(0);
        wait_vm<LDEPTH>();
        lds_write(cnt_addr, max((t >> 1) + 1 - LDEPTH, 0));
      }
      drain<LDEPTH - 1>(cnt_addr);
    }
    wait_vm<0>();
    lds_write(cnt_addr, NUNIT / 2);
    if (par == 1) STAMP(2);
    return;
  }

  float scale0 = p.sx[0] * p.sw[0];
  unsigned out_lo = (unsigned)(size_t)p.out, out_hi = (unsigned)((size_t)p.out >> 32);
  asm volatile("" : "+v"(scale0), "+v"(out_lo), "+v"(out_hi));
  if (wave == 0) {
    const bool ok = wait_flags<8>(myws + OFF_XFLAG + (bid & 7) * 256, 64, lane, 4, call, call, tmo);
    while (lds_read(ctl + CTL_GO) == 0) __builtin_amdgcn_s_sleep(1);
    STAMP(9);
    __builtin_amdgcn_fence(__ATOMIC_ACQUIRE, "");
    asm volatile("s_waitcnt vmcnt(0)" ::: "memory");
    if (lane == 0) __hip_atomic_fetch_add((GAS unsigned*)(myws + OFF_FC + (bid & 7) * 128), 1u, __ATOMIC_RELAXED, __HIP_MEMORY_SCOPE_AGENT);
    STAMP(3);
    const bool ok2 = wait_flags<2>(myws + OFF_FC + (bid & 7) * 128, 1, lane, 4, call * (NSLAB / 8), call, tmo);
    if (!ok || !ok2) lds_write(ctl + CTL_BAD, 1);
    lds_write(ctl + CTL_L2, 1);
    STAMP(10);
    if (ld_agent(tmo) == call) lds_write(ctl + CTL_BAD, 1);
  } else {
    while (lds_read(ctl + CTL_L2) == 0) __builtin_amdgcn_s_sleep(4);
  }
  asm volatile("" ::: "memory");

  const char* xa = myws + OFF_X + (size_t)(call & 1) * STAGE_BYTES + wave * 2048 + lane * 16;
  auto a_addr = [&](int t) { return xa + (size_t)((((j0 + t) & (NR - 1)) * NKS + (t >> 3)) << 14); };
  static_assert(NUNIT % PF == 0 && PF % 2 == 0, "prefetch depth");
  i32x4 a[PF][2];
#pragma unroll
  for (int i = 0; i < PF; ++i) {
    const char* q = a_addr(i);
    a[i][0] = *(const GAS i32x4*)q;
    a[i][1] = *(const GAS i32x4*)(q + 1024);
  }
  i32x16 acc0 = {}, acc1 = {};
  unsigned landed[2] = {0, 0};
  auto step = [&](int t, int i, bool prefetch) {
    while (landed[i & 1] <= (unsigned)(t >> 1)) {
      landed[i & 1] = __builtin_amdgcn_readfirstlane(lds_read(ctl + CTL_LANDED + (i & 1) * 4));
      if (landed[i & 1] <= (unsigned)(t >> 1)) __builtin_amdgcn_s_sleep(1);
    }
    const i32x4 b = *(const LAS i32x4*)(lds + t * 1024 + lane * 16);
    acc0 = __builtin_amdgcn_mfma_i32_32x32x32_i8(a[i][0], b, acc0, 0, 0, 0);
    acc1 = __builtin_amdgcn_mfma_i32_32x32x32_i8(a[i][1], b, acc1, 0, 0, 0);
    if (prefetch) {
      const char* q = a_addr(t + PF);
      a[i][0] = *(const GAS i32x4*)q;
      a[i][1] = *(const GAS i32x4*)(q + 1024);
    }
  };
  for (int t0 = 0; t0 < NUNIT - PF; t0 += PF) {
#pragma unroll
    for (int i = 0; i < PF; ++i) step(t0 + i, i, true);
  }
#pragma unroll
  for (int i = 0; i < PF; ++i) step(NUNIT - PF + i, i, false);
  if (wave == 0) STAMP(4);
  STAMP(16 + wave);

  const float scale = lds_read(ctl + CTL_BAD) != 0 ? __builtin_nanf("") : scale0;
  const int c31 = lane & 31;
  float* outp = (float*)(((size_t)(unsigned)__builtin_amdgcn_readfirstlane(out_hi) << 32) | (unsigned)__builtin_amdgcn_readfirstlane(out_lo));
  const __amdgpu_buffer_rsrc_t ors = __builtin_amdgcn_make_buffer_rsrc(outp, 0, MS * NDIM * 4, 0x00020000);
  const unsigned voff = (unsigned)(((wave * 64 + (lane >> 5) * 4) * NDIM + slab * 32 + (c31 & 7) * 4 + (c31 >> 3)) * 4);
  constexpr float LOG2E = 1.44269504088896340736f;
  const f32x2 s2 = {scale, scale}, t2 = {-scale * LOG2E, -scale * LOG2E}, one2 = {1.0f, 1.0f};
#pragma unroll
  for (int h = 0; h < 2; ++h) {
#pragma unroll
    for (int r = 0; r < 16; r += 2) {
      const f32x2 f = {(float)(h == 0 ? acc0[r] : acc1[r]), (float)(h == 0 ? acc0[r + 1] : acc1[r + 1])};
      const f32x2 y = f * s2, t = f * t2;
      const f32x2 e = {__builtin_amdgcn_exp2f(__builtin_amdgcn_fmed3f(t.x, -60.0f * LOG2E, 60.0f * LOG2E)),
                       __builtin_amdgcn_exp2f(__builtin_amdgcn_fmed3f(t.y, -60.0f * LOG2E, 60.0f * LOG2E))};
      const f32x2 d = e + one2;
      const f32x2 v = y * (f32x2){__builtin_amdgcn_rcpf(d.x), __builtin_amdgcn_rcpf(d.y)};
      const float v0 = v[0], v1 = v[1];
      __builtin_amdgcn_raw_buffer_store_b32(__float_as_uint(v0), ors, voff, (h * 32 + (r & 3) + 8 * (r >> 2)) * NDIM * 4, 17);
      __builtin_amdgcn_raw_buffer_store_b32(__float_as_uint(v1), ors, voff, (h * 32 + ((r + 1) & 3) + 8 * (r >> 2)) * NDIM * 4, 17);
    }
  }
}

}

extern "C" void kernel_launch(int rank, int nranks, ncclComm_t comm, hipStream_t stream,
                              void* const* d_in, const int* in_sizes, int n_in,
                              void* d_out, int out_size, const int* in_dtypes, int out_dtype,
                              void* const* d_out_all, void* const* d_ws_all, size_t ws_bytes) {
  (void)comm; (void)in_sizes; (void)in_dtypes; (void)out_dtype; (void)d_out_all;
  if (nranks != NR || n_in != 4 || out_size != MS * NDIM || ws_bytes < WS_NEED || rank < 0 || rank >= NR) {
    fprintf(stderr, "kernel_launch: unexpected configuration\n");
    return;
  }
  static bool attr_set[NR] = {};
  if (!attr_set[rank]) {
    HIP_CHECK(hipFuncSetAttribute((const void*)gemm_ag_kernel, hipFuncAttributeMaxDynamicSharedMemorySize, LDS_BYTES));
    attr_set[rank] = true;
  }
  Params p;
  p.x = (const char*)d_in[0];
  p.w = (const char*)d_in[1];
  p.sx = (const float*)d_in[2];
  p.sw = (const float*)d_in[3];
  p.out = (float*)d_out;
  for (int i = 0; i < NR; ++i) p.ws[i] = (char*)d_ws_all[i];
  p.myws = p.ws[rank];
  p.rank = rank;
  p.pad = 0;
  hipLaunchKernelGGL(gemm_ag_kernel, dim3(GRID), dim3(THREADS), LDS_BYTES, stream, p);
  HIP_CHECK(hipGetLastError());
}
```

```cpp
#include <hip/hip_runtime.h>
#include <rccl/rccl.h>
#include <cstdio>

#define HIP_CHECK(expr) do { hipError_t _e = (expr); if (_e != hipSuccess) { \
    fprintf(stderr, "HIP error %s at %s:%d\n", hipGetErrorString(_e), __FILE__, __LINE__); \
    return; } } while (0)

#define GAS __attribute__((address_space(1)))
#define LAS __attribute__((address_space(3)))
typedef unsigned u32x4 __attribute__((ext_vector_type(4)));
typedef int i32x4 __attribute__((ext_vector_type(4)));
typedef int i32x16 __attribute__((ext_vector_type(16)));
typedef float f32x2 __attribute__((ext_vector_type(2)));
typedef unsigned long long u64;

#define PACE16 442
#define XDELAY 90
#define XPACE 112
#define GODELAY 440
#define DELTA 60
constexpr int DEPTH = 3;
constexpr int LDEPTH = 2;
constexpr int PDEPTH = 4;
constexpr int PSKIP = 3;
constexpr int RAMP = 10;
constexpr int FALLBACK = 200;
constexpr int PF = 4;

namespace {

constexpr int NR = 8;
constexpr int NDIM = 4096;
constexpr int KS = 512;
constexpr int MS = 512;
constexpr int NSLAB = NDIM / 32;
constexpr int NKS = KS / 32;
constexpr int NUNIT = NR * NKS;
constexpr int N_WST = 64, N_STAGE = 128;
constexpr int GRID = N_STAGE + NSLAB;
constexpr int THREADS = 640;

constexpr size_t OFF_CNT = 0;
constexpr size_t OFF_TMO = OFF_CNT + GRID * 128;
constexpr size_t OFF_WCNT = OFF_TMO + 128;
constexpr size_t OFF_FC = OFF_WCNT + 8 * 128;
constexpr size_t OFF_GO = OFF_FC + 8 * 128;
constexpr size_t OFF_XFLAG = OFF_GO + 8 * 128;
constexpr size_t OFF_SW = 48 << 10;
constexpr size_t STAGE_BYTES = 2u << 20;
constexpr size_t OFF_X = 1u << 20;
constexpr size_t OFF_W = OFF_X + 2 * STAGE_BYTES;
constexpr size_t WS_NEED = OFF_W + 2 * STAGE_BYTES;
static_assert(OFF_XFLAG + 8 * 256 <= OFF_SW && OFF_SW + NSLAB * 128 <= OFF_X, "layout");

constexpr int LDS_B = NUNIT * 1024;
constexpr int LDS_CTL = LDS_B;
constexpr int CTL_LANDED = 0;
constexpr int CTL_BAD = 8;
constexpr int CTL_GO = 12;
constexpr int CTL_T0 = 16;
constexpr int CTL_L2 = 20;
constexpr int CTL_LAG = 24;
constexpr int LDS_BYTES = LDS_B + 64;

constexpr u64 DEADLINE_TICKS = 50000000ull;

struct Params {
  const char* x;
  const char* w;
  const float* sx;
  const float* sw;
  float* out;
  char* myws;
  char* ws[NR];
  int rank;
  int pad;
};

__device__ __forceinline__ u64 rt() { return __builtin_amdgcn_s_memrealtime(); }
template <int N> __device__ __forceinline__ void wait_vm() { asm volatile("s_waitcnt vmcnt(%0)" :: "i"(N) : "memory"); }
__device__ __forceinline__ void store_wt(void* p, u32x4 v) {
  asm volatile("global_store_dwordx4 %0, %1, off sc0 sc1\n\ts_nop 1" :: "v"(p), "v"(v) : "memory");
}
__device__ __forceinline__ void lds_write(unsigned addr, unsigned v) {
  asm volatile("ds_write_b32 %0, %1" :: "v"(addr), "v"(v) : "memory");
}
template <int D> __device__ __forceinline__ void drain(unsigned addr) {
  if constexpr (D > 0) {
    wait_vm<D>();
    lds_write(addr, NUNIT / 2 - D);
    drain<D - 1>(addr);
  }
}
__device__ __forceinline__ unsigned lds_read(unsigned addr) {
  unsigned v;
  asm volatile("ds_read_b32 %0, %1\n\ts_waitcnt lgkmcnt(0)" : "=v"(v) : "v"(addr) : "memory");
  return v;
}
__device__ __forceinline__ unsigned ld_sys(const void* p) {
  return __hip_atomic_load((const GAS unsigned*)p, __ATOMIC_RELAXED, __HIP_MEMORY_SCOPE_SYSTEM);
}
__device__ __forceinline__ unsigned ld_agent(const void* p) {
  return __hip_atomic_load((const GAS unsigned*)p, __ATOMIC_RELAXED, __HIP_MEMORY_SCOPE_AGENT);
}
__device__ __forceinline__ void st_sys(void* p, unsigned v) {
  __hip_atomic_store((GAS unsigned*)p, v, __ATOMIC_RELAXED, __HIP_MEMORY_SCOPE_SYSTEM);
}
__device__ __forceinline__ void st_agent(void* p, unsigned v) {
  __hip_atomic_store((GAS unsigned*)p, v, __ATOMIC_RELAXED, __HIP_MEMORY_SCOPE_AGENT);
}
__device__ __forceinline__ unsigned poll_hint(const void* p) {
  unsigned v;
  asm volatile("s_load_dword %0, %1, 0x0 glc\n\ts_waitcnt lgkmcnt(0)" : "=s"(v) : "s"(p) : "memory");
  return v;
}
__device__ __forceinline__ unsigned perm(unsigned hi, unsigned lo, unsigned sel) { return __builtin_amdgcn_perm(hi, lo, sel); }
__device__ __forceinline__ unsigned sched(int t) { return (unsigned)(t * PACE16) >> 4; }
__device__ __forceinline__ unsigned phase(int slab) { return (unsigned)(((slab * 37) & 127) * PACE16) >> 11; }
__device__ __forceinline__ void wait_until(unsigned t) {
  while ((int)((unsigned)rt() - t) < 0) __builtin_amdgcn_s_sleep(1);
}

template <int SLEEP>
__device__ __forceinline__ bool wait_flags(const char* base, int n, int lane, int stride, unsigned target, unsigned call, char* tmo) {
  const u64 t0 = rt();
  const char* q = base + (lane < n ? lane : 0) * stride;
  for (unsigned it = 1;; ++it) {
    const unsigned v = ld_sys(q);
    if (__all(v >= target)) return true;
    if ((it & 63) == 0 && (ld_agent(tmo) == call || rt() - t0 > DEADLINE_TICKS)) { st_agent(tmo, call); return false; }
    __builtin_amdgcn_s_sleep(SLEEP);
  }
}

__device__ __forceinline__ const char* frag_src(u64 tab, int t, int lane) {
  const unsigned lo = __builtin_amdgcn_readlane((unsigned)tab, t & 7), hi = __builtin_amdgcn_readlane((unsigned)(tab >> 32), t & 7);
  return (const char*)(((u64)hi << 32) | lo) + (t >> 3) * 1024 + lane * 16;
}

__global__ void __launch_bounds__(THREADS) gemm_ag_kernel(Params p) {
  extern __shared__ __attribute__((aligned(16))) char lds[];
  const int tid = threadIdx.x;
  const int wave = __builtin_amdgcn_readfirstlane(tid >> 6);
  const int lane = tid & 63;
  const int bid = blockIdx.x;
  char* const myws = p.myws;
  char* const tmo = myws + OFF_TMO;
  const unsigned ctl = (unsigned)(size_t)(LAS char*)lds + LDS_CTL;

  char* const cntp = myws + OFF_CNT + bid * 128;
  unsigned cnt_v = ld_agent(cntp);
#define GET_CALL() ({ asm volatile("" : "+v"(cnt_v) :: "memory"); (unsigned)__builtin_amdgcn_readfirstlane(cnt_v) + 1; })
  const unsigned t_start = (unsigned)rt();

  const int sb = bid ^ 64;
  if (bid < N_STAGE && sb < N_WST) {
    const int G = sb >> 1, kh = sb & 1;
    const int pslab = sb + (wave & 1) * 64, pj0 = (pslab + (pslab >> 3)) & (NR - 1);
    u64 tab = (u64)p.ws[wave == 2 ? (lane & (NR - 1)) : ((pj0 + lane) & (NR - 1))];
    unsigned call;
    if (wave < 8) {
      const int ks = kh * 8 + wave;
      unsigned r[16];
      u32x4 v[4];
      const char* src = p.w + (size_t)(ks * 32 + (lane >> 5) * 16) * NDIM + G * 128 + (lane & 31) * 4;
#pragma unroll
      for (int i = 0; i < 16; ++i) r[i] = *(const GAS unsigned*)(src + (size_t)i * NDIM);
      call = GET_CALL();
#pragma unroll
      for (int g = 0; g < 4; ++g) {
        const unsigned lo01 = perm(r[4 * g + 1], r[4 * g], 0x05010400u), hi01 = perm(r[4 * g + 1], r[4 * g], 0x07030602u);
        const unsigned lo23 = perm(r[4 * g + 3], r[4 * g + 2], 0x05010400u), hi23 = perm(r[4 * g + 3], r[4 * g + 2], 0x07030602u);
        v[0][g] = perm(lo23, lo01, 0x05040100u);
        v[1][g] = perm(lo23, lo01, 0x07060302u);
        v[2][g] = perm(hi23, hi01, 0x05040100u);
        v[3][g] = perm(hi23, hi01, 0x07060302u);
      }
      const int slab = G * 4 + ((lane >> 3) & 3);
      char* dst = myws + OFF_W + (size_t)(call & 1) * STAGE_BYTES + (size_t)(slab * NKS + ks) * 1024 + ((lane & 7) + (lane >> 5) * 32) * 16;
#pragma unroll
      for (int q = 0; q < 4; ++q) store_wt(dst + q * 128, v[q]);
      wait_vm<0>();
    } else {
      call = GET_CALL();
    }
    asm volatile("s_barrier" ::: "memory");
    if (tid == 0) {
      __hip_atomic_fetch_add((GAS unsigned*)(myws + OFF_WCNT + (bid & 7) * 128), 1u, __ATOMIC_RELAXED, __HIP_MEMORY_SCOPE_AGENT);
      st_agent(cntp, call);
    }
    if (sb == 0 && wave == 2) {
      const bool ok = wait_flags<1>(myws + OFF_WCNT, 8, lane, 128, call * (N_WST / 8), call, tmo);
      wait_until(t_start + GODELAY);
      if (ok) st_sys((char*)tab + OFF_GO + (lane >> 3) * 128 + p.rank * 4, call);
    }
    if (wave < 2) {
      tab += OFF_W + (size_t)(call & 1) * STAGE_BYTES + (size_t)pslab * NKS * 1024;
      if (!wait_flags<1>(myws + OFF_GO + (bid & 7) * 128, 8, lane, 4, call, call, tmo)) return;
      const unsigned t0 = (unsigned)rt() + phase(pslab);
      u64 m;
      {
        const u64 tw0 = rt();
        for (unsigned it = 1;; ++it) {
          m = __hip_atomic_load((const GAS u64*)(myws + OFF_SW + pslab * 128), __ATOMIC_RELAXED, __HIP_MEMORY_SCOPE_AGENT);
          if (__builtin_amdgcn_readfirstlane((unsigned)(m >> 32)) == call) break;
          if ((it & 63) == 0 && (ld_agent(tmo) == call || rt() - tw0 > DEADLINE_TICKS)) return;
          __builtin_amdgcn_s_sleep(2);
        }
      }
      const unsigned mw = __builtin_amdgcn_readfirstlane((unsigned)m);
      const unsigned tp = t0 + (mw & 0xffffu);
      for (int t = (int)(mw >> 16) + PSKIP; t < NUNIT; ++t) {
        if (((pj0 + t) & (NR - 1)) == p.rank) continue;
        const char* s = frag_src(tab, t, lane);
        wait_until(tp + sched(t));
        __builtin_amdgcn_global_load_lds((const GAS unsigned*)s, (LAS unsigned*)(lds + ((wave * 16 + (t & 15)) << 10)), 16, 0, 0);
        if (lane == 0) st_agent(myws + OFF_SW + pslab * 128 + 64, (call << 8) | (unsigned)(t + 1));
        wait_vm<2 * PDEPTH>();
      }
      wait_vm<0>();
    }
    return;
  }
  if (bid < N_STAGE) {
    const int idx = sb - N_WST;
    const int dst_rank = (p.rank + (idx >> 3)) & (NR - 1), part = idx & 7;
    char* const dws = p.ws[dst_rank];
    unsigned call = 0;
    if (wave < 8) {
      const int rtile = part * 2 + (wave & 1), ks0 = (wave >> 1) * 4;
      u32x4 v[4];
      const char* src = p.x + (size_t)(dst_rank * MS + rtile * 32 + (lane & 31)) * KS + ks0 * 32 + (lane >> 5) * 16;
#pragma unroll
      for (int i = 0; i < 4; ++i) v[i] = *(const GAS u32x4*)(src + i * 32);
      call = GET_CALL();
      char* xdst = dws + OFF_X + (size_t)(call & 1) * STAGE_BYTES + (size_t)((p.rank * NKS + ks0) * 16 + rtile) * 1024 + lane * 16;
#pragma unroll
      for (int i = 0; i < 4; ++i) {
        wait_until(t_start + XDELAY + ((wave * XPACE) >> 3) + i * XPACE);
        store_wt(xdst + i * 16384, v[i]);
      }
    }
    wait_vm<0>();
    asm volatile("s_barrier" ::: "memory");
    if (tid < 8) st_sys(dws + OFF_XFLAG + tid * 256 + (p.rank * 8 + part) * 4, call);
    if (tid == 0) st_agent(cntp, call);
    return;
  }

  const int slab = bid - N_STAGE;
  const int j0 = (slab + (slab >> 3)) & (NR - 1);
  u64 tab = 0;
  if (wave >= 8) tab = (u64)p.ws[(j0 + lane) & (NR - 1)];
  const unsigned call = GET_CALL();
  if (tid < 16) lds_write(ctl + tid * 4, 0);
  asm volatile("s_waitcnt lgkmcnt(0)\n\ts_barrier" ::: "memory");
  if (tid == 0) st_agent(cntp, call);

  if (wave >= 8) {
    const int par = wave - 8;
    const unsigned cnt_addr = ctl + CTL_LANDED + par * 4;
    tab += OFF_W + (size_t)(call & 1) * STAGE_BYTES + (size_t)slab * NKS * 1024;
    unsigned t0;
    if (par == 0) {
      const bool ok = wait_flags<1>(myws + OFF_GO + (bid & 7) * 128, 8, lane, 4, call, call, tmo);
      t0 = (unsigned)rt() + phase(slab);
      if (!ok) {
        lds_write(ctl + CTL_BAD, 1);
        lds_write(ctl + CTL_T0, 0);
        lds_write(ctl + CTL_LANDED, NUNIT / 2);
        lds_write(ctl + CTL_LANDED + 4, NUNIT / 2);
        lds_write(ctl + CTL_GO, 2);
        if (lane == 0)
          __hip_atomic_store((GAS u64*)(myws + OFF_SW + slab * 128), ((u64)call << 32) | ((unsigned)NUNIT << 16), __ATOMIC_RELAXED, __HIP_MEMORY_SCOPE_AGENT);
        return;
      }
      lds_write(ctl + CTL_T0, t0);
      lds_write(ctl + CTL_GO, 1);
    } else {
      unsigned g;
      while ((g = __builtin_amdgcn_readfirstlane(lds_read(ctl + CTL_GO))) == 0) __builtin_amdgcn_s_sleep(1);
      if (g == 2) return;
      t0 = __builtin_amdgcn_readfirstlane(lds_read(ctl + CTL_T0));
    }
    int t = par;
    unsigned lag1 = 0;
    for (; t < NUNIT; t += 2) {
      const char* s = frag_src(tab, t, lane);
      wait_until(t0 + sched(t));
      lag1 = __builtin_amdgcn_readfirstlane(lds_read(ctl + (par == 0 ? CTL_L2 : CTL_LAG)));
      if (lag1) break;
      __builtin_amdgcn_global_load_lds((const GAS unsigned*)s, (LAS unsigned*)(lds + t * 1024), 16, 0, 17);
      wait_vm<DEPTH>();
      lds_write(cnt_addr, max((t >> 1) + 1 - DEPTH, 0));
    }
    if (par == 0) {
      unsigned lag = (unsigned)rt() - (t0 + sched(t));
      lag = lag < 0xfffeu ? lag : 0xfffeu;
      if (lane == 0)
        __hip_atomic_store((GAS u64*)(myws + OFF_SW + slab * 128), ((u64)call << 32) | ((unsigned)t << 16) | lag, __ATOMIC_RELAXED, __HIP_MEMORY_SCOPE_AGENT);
      lag1 = lag + 1;
      if (t < NUNIT) lds_write(ctl + CTL_LAG, lag1);
    }
    if (t < NUNIT) {
      const unsigned tl = t0 + lag1 - 1, te = sched(NUNIT - 1);
      const char* prog = myws + OFF_SW + slab * 128 + 64;
      const int t_first = t + PSKIP + 1;
      unsigned v = 0, delta = 0;
      for (; t < NUNIT; t += 2) {
        const char* s = frag_src(tab, t, lane);
        const unsigned ts = sched(t) + delta;
        delta = delta + 2 * RAMP < DELTA ? delta + 2 * RAMP : DELTA;
        const unsigned target = (call << 8) | (unsigned)(t + 1);
        const bool follow = t >= t_first && ((j0 + t) & (NR - 1)) != p.rank;
        if (follow && v < target) v = poll_hint(prog);
        wait_until(tl + (ts < te ? ts : te));
        if (follow) {
          unsigned tchg = (unsigned)rt();
          while (v < target) {
            const unsigned w = poll_hint(prog), now = (unsigned)rt();
            if (w != v) { v = w; tchg = now; }
            else if (now - tchg > FALLBACK) break;
          }
        }
        __builtin_amdgcn_s_setprio(3);
        __builtin_amdgcn_global_load_lds((const GAS unsigned*)s, (LAS unsigned*)(lds + t * 1024), 16, 0, 0);
        __builtin_amdgcn_s_setprio(0);
        wait_vm<LDEPTH>();
        lds_write(cnt_addr, max((t >> 1) + 1 - LDEPTH, 0));
      }
      drain<LDEPTH - 1>(cnt_addr);
    }
    wait_vm<0>();
    lds_write(cnt_addr, NUNIT / 2);
    return;
  }

  float scale0 = p.sx[0] * p.sw[0];
  unsigned out_lo = (unsigned)(size_t)p.out, out_hi = (unsigned)((size_t)p.out >> 32);
  asm volatile("" : "+v"(scale0), "+v"(out_lo), "+v"(out_hi));
  if (wave == 0) {
    const bool ok = wait_flags<8>(myws + OFF_XFLAG + (bid & 7) * 256, 64, lane, 4, call, call, tmo);
    while (lds_read(ctl + CTL_GO) == 0) __builtin_amdgcn_s_sleep(1);
    __builtin_amdgcn_fence(__ATOMIC_ACQUIRE, "");
    asm volatile("s_waitcnt vmcnt(0)" ::: "memory");
    if (lane == 0) __hip_atomic_fetch_add((GAS unsigned*)(myws + OFF_FC + (bid & 7) * 128), 1u, __ATOMIC_RELAXED, __HIP_MEMORY_SCOPE_AGENT);
    const bool ok2 = wait_flags<2>(myws + OFF_FC + (bid & 7) * 128, 1, lane, 4, call * (NSLAB / 8), call, tmo);
    if (!ok || !ok2) lds_write(ctl + CTL_BAD, 1);
    lds_write(ctl + CTL_L2, 1);
    if (ld_agent(tmo) == call) lds_write(ctl + CTL_BAD, 1);
  } else {
    while (lds_read(ctl + CTL_L2) == 0) __builtin_amdgcn_s_sleep(4);
  }
  asm volatile("" ::: "memory");

  const char* xa = myws + OFF_X + (size_t)(call & 1) * STAGE_BYTES + wave * 2048 + lane * 16;
  auto a_addr = [&](int t) { return xa + (size_t)((((j0 + t) & (NR - 1)) * NKS + (t >> 3)) << 14); };
  static_assert(NUNIT % PF == 0 && PF % 2 == 0, "prefetch depth");
  i32x4 a[PF][2];
#pragma unroll
  for (int i = 0; i < PF; ++i) {
    const char* q = a_addr(i);
    a[i][0] = *(const GAS i32x4*)q;
    a[i][1] = *(const GAS i32x4*)(q + 1024);
  }
  i32x16 acc0 = {}, acc1 = {};
  unsigned landed[2] = {0, 0};
  auto step = [&](int t, int i, bool prefetch) {
    while (landed[i & 1] <= (unsigned)(t >> 1)) {
      landed[i & 1] = __builtin_amdgcn_readfirstlane(lds_read(ctl + CTL_LANDED + (i & 1) * 4));
      if (landed[i & 1] <= (unsigned)(t >> 1)) __builtin_amdgcn_s_sleep(1);
    }
    const i32x4 b = *(const LAS i32x4*)(lds + t * 1024 + lane * 16);
    acc0 = __builtin_amdgcn_mfma_i32_32x32x32_i8(a[i][0], b, acc0, 0, 0, 0);
    acc1 = __builtin_amdgcn_mfma_i32_32x32x32_i8(a[i][1], b, acc1, 0, 0, 0);
    if (prefetch) {
      const char* q = a_addr(t + PF);
      a[i][0] = *(const GAS i32x4*)q;
      a[i][1] = *(const GAS i32x4*)(q + 1024);
    }
  };
  for (int t0 = 0; t0 < NUNIT - PF; t0 += PF) {
#pragma unroll
    for (int i = 0; i < PF; ++i) step(t0 + i, i, true);
  }
#pragma unroll
  for (int i = 0; i < PF; ++i) step(NUNIT - PF + i, i, false);

  const float scale = lds_read(ctl + CTL_BAD) != 0 ? __builtin_nanf("") : scale0;
  const int c31 = lane & 31;
  float* outp = (float*)(((size_t)(unsigned)__builtin_amdgcn_readfirstlane(out_hi) << 32) | (unsigned)__builtin_amdgcn_readfirstlane(out_lo));
  const __amdgpu_buffer_rsrc_t ors = __builtin_amdgcn_make_buffer_rsrc(outp, 0, MS * NDIM * 4, 0x00020000);
  const unsigned voff = (unsigned)(((wave * 64 + (lane >> 5) * 4) * NDIM + slab * 32 + (c31 & 7) * 4 + (c31 >> 3)) * 4);
  constexpr float LOG2E = 1.44269504088896340736f;
  const f32x2 s2 = {scale, scale}, t2 = {-scale * LOG2E, -scale * LOG2E}, one2 = {1.0f, 1.0f};
#pragma unroll
  for (int h = 0; h < 2; ++h) {
#pragma unroll
    for (int r = 0; r < 16; r += 2) {
      const f32x2 f = {(float)(h == 0 ? acc0[r] : acc1[r]), (float)(h == 0 ? acc0[r + 1] : acc1[r + 1])};
      const f32x2 y = f * s2, t = f * t2;
      const f32x2 e = {__builtin_amdgcn_exp2f(__builtin_amdgcn_fmed3f(t.x, -60.0f * LOG2E, 60.0f * LOG2E)),
                       __builtin_amdgcn_exp2f(__builtin_amdgcn_fmed3f(t.y, -60.0f * LOG2E, 60.0f * LOG2E))};
      const f32x2 d = e + one2;
      const f32x2 v = y * (f32x2){__builtin_amdgcn_rcpf(d.x), __builtin_amdgcn_rcpf(d.y)};
      const float v0 = v[0], v1 = v[1];
      __builtin_amdgcn_raw_buffer_store_b32(__float_as_uint(v0), ors, voff, (h * 32 + (r & 3) + 8 * (r >> 2)) * NDIM * 4, 17);
      __builtin_amdgcn_raw_buffer_store_b32(__float_as_uint(v1), ors, voff, (h * 32 + ((r + 1) & 3) + 8 * (r >> 2)) * NDIM * 4, 17);
    }
  }
}

}

extern "C" void kernel_launch(int rank, int nranks, ncclComm_t comm, hipStream_t stream,
                              void* const* d_in, const int* in_sizes, int n_in,
                              void* d_out, int out_size, const int* in_dtypes, int out_dtype,
                              void* const* d_out_all, void* const* d_ws_all, size_t ws_bytes) {
  (void)comm; (void)in_sizes; (void)in_dtypes; (void)out_dtype; (void)d_out_all;
  if (nranks != NR || n_in != 4 || out_size != MS * NDIM || ws_bytes < WS_NEED || rank < 0 || rank >= NR) {
    fprintf(stderr, "kernel_launch: unexpected configuration\n");
    return;
  }
  static bool attr_set[NR] = {};
  if (!attr_set[rank]) {
    HIP_CHECK(hipFuncSetAttribute((const void*)gemm_ag_kernel, hipFuncAttributeMaxDynamicSharedMemorySize, LDS_BYTES));
    attr_set[rank] = true;
  }
  Params p;
  p.x = (const char*)d_in[0];
  p.w = (const char*)d_in[1];
  p.sx = (const float*)d_in[2];
  p.sw = (const float*)d_in[3];
  p.out = (float*)d_out;
  for (int i = 0; i < NR; ++i) p.ws[i] = (char*)d_ws_all[i];
  p.myws = p.ws[rank];
  p.rank = rank;
  p.pad = 0;
  hipLaunchKernelGGL(gemm_ag_kernel, dim3(GRID), dim3(THREADS), LDS_BYTES, stream, p);
  HIP_CHECK(hipGetLastError());
}
```
